```python
import numpy as np

kernel_name = "gpu_a2a_gemm_m4096_k4096_n4096_e4m3e4m3_dqsilu"


ABSMAX_RTOL = 0.03

COMPUTE_DTYPE = "float32"
EPILOGUE = "dequant_silu"
IN_QUANT = None
OUT_QUANT = None
IN_FP8 = "e4m3e4m3"
IN_DTYPES = {"x": "e4m3", "w_mat": "e4m3",
             "scale_x": "float32", "scale_w": "float32"}
WORLD_SIZE = 8

def _snap_fp8(x, mant_bits, max_abs, min_normal):
    a = np.abs(x.astype(np.float64))
    _m, _e = np.frexp(a)
    q = np.where(a >= min_normal, np.ldexp(1.0, _e - 1 - mant_bits),
                 min_normal * 2.0 ** -mant_bits)
    snapped = np.minimum(np.round(a / q) * q, max_abs)
    return (np.sign(x) * snapped).astype(np.float32)

def setup_inputs():

    x = _snap_fp8(np.random.randn(4096, 4096).astype(np.float32), 3, 448.0, 2.0 ** -6)
    w_mat = _snap_fp8(np.random.randn(4096, 4096).astype(np.float32), 3, 448.0, 2.0 ** -6)
    scale_x = (0.02 * (1.0 + np.random.rand(1))).astype(np.float32)
    scale_w = (0.02 * (1.0 + np.random.rand(1))).astype(np.float32)
    return {"x": x, "w_mat": w_mat, "scale_x": scale_x, "scale_w": scale_w}

def reference(x, w_mat, scale_x, scale_w):
    acc = (x @ w_mat).astype(np.float32)
    y = (acc * (scale_x * scale_w)).astype(np.float32)
    return (y / (1.0 + np.exp(-np.clip(y, -60.0, 60.0)))).astype(np.float32)
```

```cpp
#include <hip/hip_runtime.h>
#include <rccl/rccl.h>
#include <cstdio>
#include <cstdlib>

#define GAS __attribute__((address_space(1)))
#define LAS __attribute__((address_space(3)))
typedef unsigned v4u __attribute__((ext_vector_type(4)));
typedef unsigned v2u __attribute__((ext_vector_type(2)));
typedef int v8i __attribute__((ext_vector_type(8)));
typedef int v2i __attribute__((ext_vector_type(2)));
typedef float v16f __attribute__((ext_vector_type(16)));
typedef float v4f __attribute__((ext_vector_type(4)));

constexpr int NRANK = 8;
constexpr int RSLOTS = 9;
constexpr int STAGE_B = 8192;
constexpr int RING_B = RSLOTS * STAGE_B;
constexpr int SCR_OFF = 2 * RING_B;
constexpr int SYNC_OFF = SCR_OFF + 8192;
constexpr int LDS_B = SYNC_OFF + 256;
constexpr size_t CTR_OFF = 0;
constexpr size_t STG_OFF = 65536;
constexpr size_t STG_B = 2u << 20;
constexpr size_t WS_NEED = STG_OFF + 3 * STG_B;
constexpr unsigned long long DEADLINE = 50000000ull;
#define STAMP(i) do {} while (0)

struct Params {
  const unsigned char* x;
  const unsigned char* w;
  const float* sx;
  const float* sw;
  float* out;
  unsigned char* ws[NRANK];
  int rank;
  int delta;
  int tstart;
  int qmax;
  int pwin;
  int tail;
};

__device__ __forceinline__ unsigned long long rt() { return __builtin_amdgcn_s_memrealtime(); }
__device__ __forceinline__ unsigned nzfix(unsigned x) {
  unsigned t = (x & 0x7f7f7f7fu) + 0x7f7f7f7fu;
  return x | (~(t | x) & 0x80808080u);
}
__device__ __forceinline__ int kidx_of(int t, int rank) {
  int tt = t - 8;
  int j = (tt * 37) >> 8;
  int pi = tt - 7 * j;
  int rem = ((rank + 1 + pi) & 7) * 8 + j;
  return t < 8 ? rank * 8 + t : rem;
}
__device__ __forceinline__ unsigned umin3(unsigned a, unsigned b, unsigned c) { return min(min(a, b), c); }
__device__ __forceinline__ bool frag_ok(const v4u& lo, const v4u& hi) {
  unsigned m = umin3(umin3(lo.x, lo.y, lo.z), umin3(lo.w, hi.x, hi.y), min(hi.z, hi.w));
  return __builtin_amdgcn_ballot_w64(m == 0u) == 0ull;
}
#define CFENCE() asm volatile("" ::: "memory")
__device__ __forceinline__ void lds_pub(unsigned addr, unsigned val) {
  asm volatile("s_mov_b64 exec, 1\n\tds_write_b32 %0, %1\n\ts_mov_b64 exec, -1" :: "v"(addr), "v"(val) : "memory");
}
#define AUX_SYS 17

struct ASet { v4u lo[4], hi[4]; };

__global__ void __launch_bounds__(512, 2) fused_a2a_gemm(Params p) {
  extern __shared__ __attribute__((aligned(16))) unsigned char lds_raw[];
  LAS unsigned char* L8 = (LAS unsigned char*)lds_raw;
  volatile LAS unsigned* SY = (volatile LAS unsigned*)(L8 + SYNC_OFF);
  const int b = blockIdx.x;
  const int wid = __builtin_amdgcn_readfirstlane(threadIdx.x >> 6), lane = threadIdx.x & 63;
  const int rank = p.rank & 7;
  const unsigned long long t0 = rt();
  unsigned char* myws = p.ws[rank];
  GAS unsigned* ctr = (GAS unsigned*)(myws + CTR_OFF + b * 64 + wid * 4);
  const int xcd = b & 7, idx = b >> 3;
  const int nt = xcd * 4 + (idx & 3), mt = idx >> 2;

  if (wid == 0) {
    __builtin_amdgcn_s_setprio(3);
    const int di = (b >> 3) & 7, rg = (b & 7) + 8 * (b >> 6);
    const int d = (rank + di) & 7;
    unsigned callv;
    {
      const unsigned char* src = p.x + (size_t)(d * 512 + rg * 16) * 512;
      unsigned voff = (lane >> 5) * 512 + ((lane & 31) ^ (lane >> 5)) * 16;
      asm volatile("global_load_dword %0, %1, off sc1" : "=v"(callv) : "v"(ctr) : "memory");
#pragma unroll
      for (int i = 0; i < 8; i++) {
        unsigned vo = (voff ^ (i * 32)) + i * 1024;
        unsigned lo = SCR_OFF + i * 1024;
        asm volatile("s_mov_b32 m0, %2\n\ts_nop 0\n\tglobal_load_lds_dwordx4 %0, %1" :: "v"(vo), "s"(src), "s"(lo) : "memory");
      }
    }
    __builtin_amdgcn_s_barrier();
    asm volatile("s_waitcnt vmcnt(0)" : "+v"(callv) :: "memory");
    const unsigned call = __builtin_amdgcn_readfirstlane(callv) + 1u;
    const int q = lane >> 4, r16 = lane & 15;
    GAS unsigned char* dst = (GAS unsigned char*)p.ws[d] + STG_OFF + (call % 3u) * STG_B + (size_t)rank * 262144 + (rg >> 1) * 2048 +
                             (q & 1) * 1024 + (q >> 1) * 512 + ((rg & 1) * 16 + r16) * 16;
    v4u v[8];
#pragma unroll
    for (int j = 0; j < 8; j++) v[j] = *(const LAS v4u*)(L8 + SCR_OFF + r16 * 512 + (((4 * j + q) ^ r16) * 16));
    const unsigned long long tb = t0 + p.tstart + (unsigned)(p.delta * rg) / 32u;
#pragma unroll
    for (int j = 0; j < 8; j++) {
      v4u y = {nzfix(v[j].x), nzfix(v[j].y), nzfix(v[j].z), nzfix(v[j].w)};
      if (di != 0) {
        while ((long long)(rt() - (tb + (unsigned)(j * p.delta))) < 0) __builtin_amdgcn_s_sleep(1);
      }
      asm volatile("global_store_dwordx4 %0, %1, off sc0 sc1\n\ts_nop 1" :: "v"(dst + j * 32768), "v"(y) : "memory");
      if (j == 0) STAMP(1);
    }
    STAMP(2);
    GAS unsigned char* z = (GAS unsigned char*)myws + STG_OFF + ((call + 2u) % 3u) * STG_B + b * 8192 + lane * 16;
    v4u zero = {0, 0, 0, 0};
#pragma unroll
    for (int j = 0; j < 8; j++)
      asm volatile("global_store_dwordx4 %0, %1, off sc0 sc1\n\ts_nop 1" :: "v"(z + j * 1024), "v"(zero) : "memory");
    if (lane == 0) __hip_atomic_store(ctr, call, __ATOMIC_RELAXED, __HIP_MEMORY_SCOPE_AGENT);
    return;
  }
  if (wid == 1) {
    SY[lane] = 0u;
    unsigned callv = __hip_atomic_load(ctr, __ATOMIC_RELAXED, __HIP_MEMORY_SCOPE_AGENT);
    asm volatile("s_waitcnt lgkmcnt(0)" ::: "memory");
    __builtin_amdgcn_s_barrier();
    const unsigned call = __builtin_amdgcn_readfirstlane(callv) + 1u;
    if (lane == 0) __hip_atomic_store(ctr, call, __ATOMIC_RELAXED, __HIP_MEMORY_SCOPE_AGENT);
    const GAS unsigned* a = (const GAS unsigned*)(myws + STG_OFF + (call % 3u) * STG_B + kidx_of(lane, rank) * 32768 +
                                                  (2 * mt + 1) * 2048 + 1024 + 512 + 256 + 240);
    unsigned long long seen = 0;
    int avail = 0;
    while (avail < 64) {
      unsigned v = 0;
      if (lane >= avail && lane < avail + p.pwin && !((seen >> lane) & 1))
        v = __hip_atomic_load(a, __ATOMIC_RELAXED, __HIP_MEMORY_SCOPE_SYSTEM);
      seen |= __builtin_amdgcn_ballot_w64(v != 0u);
      int na = (~seen == 0ull) ? 64 : __builtin_ctzll(~seen);
      if (na > avail) {
        avail = na;
        lds_pub(SYNC_OFF + 4, (unsigned)avail);
        lds_pub(SYNC_OFF + 12, (unsigned)avail);
      } else {
        if (rt() - t0 > DEADLINE) break;
        __builtin_amdgcn_s_sleep(2);
      }
    }
    return;
  }
  if (wid < 4) {
    const int kp = wid - 2;
    const unsigned voff = (lane >> 3) * 4096 + (((lane & 7) ^ ((lane >> 3) & 6)) * 16);
    const unsigned char* wbase = p.w + nt * 128;
    __builtin_amdgcn_s_barrier();
    int issued = 0, landed = 0, freed = 0, slot = 0;
    while (landed < 32) {
      bool act = false;
      if (issued < 32 && issued - landed < p.qmax && issued < freed + RSLOTS) {
        const unsigned char* src = wbase + (size_t)kidx_of(2 * issued + kp, rank) * (64 * 4096);
        unsigned lo = kp * RING_B + slot * STAGE_B;
#pragma unroll
        for (int i = 0; i < 8; i++) {
          asm volatile("s_mov_b32 m0, %2\n\ts_nop 0\n\tglobal_load_lds_dwordx4 %0, %1"
                       :: "v"(voff), "s"(src + i * 32768), "s"(lo + i * 1024) : "memory");
        }
        issued++;
        slot = slot == RSLOTS - 1 ? 0 : slot + 1;
        act = true;
      }
      CFENCE();
      unsigned ib = __builtin_amdgcn_s_getreg(7 | (31 << 11));
      CFENCE();
      int vm = (ib & 15) | ((ib >> 18) & 48);
      int l = issued - ((vm + 7) >> 3);
      if (l > landed) {
        landed = l;
        lds_pub(SYNC_OFF + 8 * kp, (unsigned)landed);
        act = true;
      }
      if (issued < 32 && issued >= freed + RSLOTS) {
        v2u dn = *(volatile LAS v2u*)(SY + 4 + 2 * kp);
        int nf = __builtin_amdgcn_readfirstlane(min(dn.x, dn.y));
        if (nf > freed) { freed = nf; act = true; }
      }
      if (!act) {
        if (rt() - t0 > DEADLINE) break;
        __builtin_amdgcn_s_sleep(1);
      }
    }
    STAMP(2);
    return;
  }

  const int cw = wid - 4, mh = cw & 1, kp = cw >> 1;
  unsigned callv = __hip_atomic_load(ctr, __ATOMIC_RELAXED, __HIP_MEMORY_SCOPE_AGENT);
  const float scale = p.sx[0] * p.sw[0];
  __builtin_amdgcn_s_barrier();
  const unsigned call = __builtin_amdgcn_readfirstlane(callv) + 1u;
  if (lane == 0) __hip_atomic_store(ctr, call, __ATOMIC_RELAXED, __HIP_MEMORY_SCOPE_AGENT);
  const auto rsA = __builtin_amdgcn_make_buffer_rsrc((void*)(myws + STG_OFF + (call % 3u) * STG_B + (2 * mt + mh) * 2048), 0,
                                                     (int)STG_B, 0x00020000);
  const int avoff = lane * 16;
  const int bq = (lane & 15) >> 1;
  const unsigned baddr0 = kp * RING_B + ((lane >> 5) * 32 + bq) * 128 + (((bq & 6) + ((lane >> 4) & 1)) * 16) + (lane & 1) * 8;
  volatile LAS v2u* SYP = (volatile LAS v2u*)(SY + 2 * kp);
  const unsigned DONE = SYNC_OFF + 4 * (4 + 2 * kp + mh);
  bool bad = false;

  v16f acc[4];
#pragma unroll
  for (int i = 0; i < 4; i++)
#pragma unroll
    for (int j = 0; j < 16; j++) acc[i][j] = 0.f;
  v8i B[2][4];
  ASet A[2];
  int landed_c = 0, avail_c = 0;
  unsigned slot_off = 0;

  auto load_frag = [&](ASet& S, int f, int g) {
    int so = kidx_of(8 * g + 2 * f + kp, rank) * 32768;
    S.lo[f] = __builtin_amdgcn_raw_buffer_load_b128(rsA, avoff, so, AUX_SYS);
    S.hi[f] = __builtin_amdgcn_raw_buffer_load_b128(rsA, avoff + 1024, so, AUX_SYS);
  };
  auto load_group = [&](ASet& S, int g) {
#pragma unroll
    for (int f = 0; f < 4; f++) load_frag(S, f, g);
  };
  auto read_B = [&](v8i (&Bs)[4]) {
    unsigned a0 = baddr0 + slot_off;
#pragma unroll
    for (int nb = 0; nb < 4; nb++) {
      unsigned a = a0 ^ (nb << 5);
#pragma unroll
      for (int r = 0; r < 4; r++) {
        v2i t = __builtin_amdgcn_ds_read_tr8_b64_v2i32((LAS v2i*)(L8 + a + r * 1024));
        Bs[nb][2 * r] = t.x;
        Bs[nb][2 * r + 1] = t.y;
      }
    }
    slot_off = slot_off == (unsigned)(RING_B - STAGE_B) ? 0u : slot_off + STAGE_B;
  };
  auto refresh = [&]() {
    v2u s = *SYP;
    landed_c = __builtin_amdgcn_readfirstlane(s.x);
    avail_c = __builtin_amdgcn_readfirstlane(s.y);
  };
  auto wait_landed = [&](int n) {
    while (landed_c < n) {
      refresh();
      if (landed_c >= n) break;
      if (rt() - t0 > DEADLINE) { bad = true; break; }
      __builtin_amdgcn_s_sleep(1);
    }
  };
  auto wait_avail = [&](int n) {
    while (avail_c < n) {
      refresh();
      if (avail_c >= n) break;
      if (rt() - t0 > DEADLINE) { bad = true; break; }
      __builtin_amdgcn_s_sleep(1);
    }
  };
  auto need_of = [&](int g) { return g == 7 ? p.tail + kp : 8 * g + 7 + kp; };

  auto group = [&](ASet& S, ASet& Nx, int g) {
    bool ok[4];
#pragma unroll
    for (int f = 0; f < 4; f++) ok[f] = frag_ok(S.lo[f], S.hi[f]);
    STAMP(1 + g);
    bool issued = g == 7;
    const int need = need_of(g + 1);
    v2u snap = *SYP;
#pragma unroll
    for (int s = 0; s < 4; s++) {
      const int u = 4 * g + s;
      landed_c = __builtin_amdgcn_readfirstlane(snap.x);
      avail_c = __builtin_amdgcn_readfirstlane(snap.y);
      if (!issued && avail_c >= need) { CFENCE(); load_group(Nx, g + 1); issued = true; }
      if (__builtin_expect(!ok[s], 0)) {
        for (;;) {
          CFENCE();
#pragma unroll
          for (int f = 0; f < 4; f++) if (!ok[f]) load_frag(S, f, g);
#pragma unroll
          for (int f = 0; f < 4; f++) ok[f] = frag_ok(S.lo[f], S.hi[f]);
          if (ok[s]) break;
          if (rt() - t0 > DEADLINE) { bad = true; break; }
        }
      }
      wait_landed(min(u + 2, 32));
      CFENCE();
      snap = *SYP;
      read_B(B[(s + 1) & 1]);
      CFENCE();
      lds_pub(DONE, (unsigned)(u + 2));
      v8i a;
      a[0] = S.lo[s].x; a[1] = S.lo[s].y; a[2] = S.lo[s].z; a[3] = S.lo[s].w;
      a[4] = S.hi[s].x; a[5] = S.hi[s].y; a[6] = S.hi[s].z; a[7] = S.hi[s].w;
#pragma unroll
      for (int nb = 0; nb < 4; nb++)
        acc[nb] = __builtin_amdgcn_mfma_scale_f32_32x32x64_f8f6f4(a, B[s & 1][nb], acc[nb], 0, 0, 0, 0, 0, 0);
    }
    if (!issued) { wait_avail(need); CFENCE(); load_group(Nx, g + 1); }
  };

  wait_avail(need_of(0));
  CFENCE();
  load_group(A[0], 0);
  wait_landed(1);
  CFENCE();
  read_B(B[0]);
  CFENCE();
  lds_pub(DONE, 1u);
#pragma unroll 1
  for (int g = 0; g < 8; g += 2) {
    group(A[0], A[1], g);
    group(A[1], A[0], g + 1);
  }
  STAMP(9);

  if (bad) {
#pragma unroll
    for (int i = 0; i < 4; i++)
#pragma unroll
      for (int j = 0; j < 16; j++) acc[i][j] = __builtin_nanf("");
  }
  {
    volatile LAS unsigned* PD = SY + 4 + 2 * kp + (mh ^ 1);
    for (;;) {
      int dn = __builtin_amdgcn_readfirstlane(*PD);
      if (dn >= 25) break;
      if (rt() - t0 > DEADLINE) break;
      __builtin_amdgcn_s_sleep(1);
    }
    CFENCE();
    LAS v4f* sc = (LAS v4f*)(L8 + kp * RING_B + ((23 + mh) % RSLOTS) * STAGE_B) + lane;
#pragma unroll
    for (int h = 0; h < 2; h++) {
#pragma unroll
      for (int j = 0; j < 4; j++) {
        v4f t;
        if (kp == 0) t = (v4f){acc[2 + h][4 * j], acc[2 + h][4 * j + 1], acc[2 + h][4 * j + 2], acc[2 + h][4 * j + 3]};
        else t = (v4f){acc[h][4 * j], acc[h][4 * j + 1], acc[h][4 * j + 2], acc[h][4 * j + 3]};
        sc[(h * 4 + j) * 64] = t;
      }
    }
    CFENCE();
    lds_pub(SYNC_OFF + 4 * (8 + cw), 1u);
    volatile LAS unsigned* RF = SY + 8 + (cw ^ 2);
    for (;;) {
      int f = __builtin_amdgcn_readfirstlane(*RF);
      if (f) break;
      if (rt() - t0 > DEADLINE) { bad = true; break; }
    }
    CFENCE();
    const LAS v4f* oc = (const LAS v4f*)(L8 + (kp ^ 1) * RING_B + ((23 + mh) % RSLOTS) * STAGE_B) + lane;
    GAS float* o = (GAS float*)p.out + (size_t)(mt * 64 + mh * 32 + (lane >> 5) * 4) * 4096 + nt * 128 + kp * 64 + (lane & 31);
    const float nl2e = -1.4426950408889634f;
#pragma unroll
    for (int h = 0; h < 2; h++) {
#pragma unroll
      for (int j = 0; j < 4; j++) {
        v4f t = oc[(h * 4 + j) * 64];
#pragma unroll
        for (int e = 0; e < 4; e++) {
          float mine = kp == 0 ? acc[h][4 * j + e] : acc[2 + h][4 * j + e];
          float y = (mine + t[e]) * scale;
          float c = __builtin_amdgcn_fmed3f(y, -60.f, 60.f);
          float ex = __builtin_amdgcn_exp2f(c * nl2e);
          float r = y * __builtin_amdgcn_rcpf(1.0f + ex);
          if (bad) r = __builtin_nanf("");
          o[(size_t)(8 * j + e) * 4096 + h * 32] = r;
        }
      }
    }
  }
}

static int envi(const char*, int d) { return d; }

extern "C" void kernel_launch(int rank, int nranks, ncclComm_t comm,
                              hipStream_t stream,
                              void* const* d_in, const int* in_sizes, int n_in,
                              void* d_out, int out_size,
                              const int* in_dtypes, int out_dtype,
                              void* const* d_out_all,
                              void* const* d_ws_all, size_t ws_bytes) {
  if (nranks != NRANK || (unsigned)rank >= (unsigned)NRANK || ws_bytes < WS_NEED || n_in < 4) {
    fprintf(stderr, "unsupported configuration\n");
    return;
  }
  Params p;
  p.x = (const unsigned char*)d_in[0];
  p.w = (const unsigned char*)d_in[1];
  p.sx = (const float*)d_in[2];
  p.sw = (const float*)d_in[3];
  p.out = (float*)d_out;
  for (int i = 0; i < NRANK; i++) p.ws[i] = (unsigned char*)d_ws_all[i];
  p.rank = rank;
  p.delta = envi("DELTA", 64);
  p.tstart = envi("TSTART", 60);
  p.qmax = envi("QMAX", 4);
  p.pwin = envi("PWIN", 10);
  p.tail = envi("TAIL", 63);
  hipError_t e = hipFuncSetAttribute((const void*)fused_a2a_gemm, hipFuncAttributeMaxDynamicSharedMemorySize, LDS_B);
  if (e != hipSuccess) fprintf(stderr, "attr: %s\n", hipGetErrorString(e));
  hipLaunchKernelGGL(fused_a2a_gemm, dim3(256), dim3(512), LDS_B, stream, p);
}
```

```cpp
#include <hip/hip_runtime.h>
#include <rccl/rccl.h>
#include <cstdio>
#include <cstdlib>

#define GAS __attribute__((address_space(1)))
#define LAS __attribute__((address_space(3)))
typedef unsigned v4u __attribute__((ext_vector_type(4)));
typedef float v16f __attribute__((ext_vector_type(16)));
typedef float v2f __attribute__((ext_vector_type(2)));

constexpr int NRANK = 8;
constexpr int STAGE_B = 8192;
constexpr int RING_B = 8 * STAGE_B;
constexpr int SCR_OFF = 2 * RING_B;
constexpr int SYNC_OFF = SCR_OFF + 8192;
constexpr int XTR_OFF = SYNC_OFF + 1024;
constexpr int LDS_B = XTR_OFF + 2 * STAGE_B;
constexpr size_t CTR_OFF = 0;
constexpr size_t FLG_OFF = 65536;
constexpr size_t STG_OFF = 2u << 20;
constexpr size_t STG_B = 2u << 20;
constexpr size_t PRT_OFF = 8u << 20;
constexpr size_t WS_NEED = PRT_OFF + (8u << 20);
constexpr unsigned long long DEADLINE = 50000000ull;

constexpr int T_SLOT = 16;
constexpr int T_TSTART = 40;
constexpr int T_LAT = 150;
constexpr int T_TMIN = 280;
constexpr int T_HOLD = 8;

struct Params {
  const unsigned char* x;
  const unsigned char* w;
  unsigned char* myws;
  int rank;
  const float* sx;
  const float* sw;
  float* out;
  unsigned char* wsr[NRANK];
};

__device__ __forceinline__ unsigned long long rt() { return __builtin_amdgcn_s_memrealtime(); }
__device__ __forceinline__ unsigned nzfix(unsigned x) {
  unsigned t = (x & 0x7f7f7f7fu) + 0x7f7f7f7fu;
  return x | (~(t | x) & 0x80808080u);
}
__device__ __forceinline__ int slot_of(int f, int mt) {
  int r = f - 4;
  int jp = (r * 37) >> 8;
  int i = r - 7 * jp;
  return jp * 8 + i + (i >= 2 * mt ? 1 : 0);
}
__device__ __forceinline__ int kidx_of(int f, int kp, int rank, int mt) {
  int s = slot_of(f, mt);
  int di = (((s >> 1) - mt) & 3) + 4 * (s & 1);
  int rem = ((rank - di) & 7) * 8 + (s >> 3) * 2 + kp;
  return f < 4 ? rank * 8 + kp * 4 + f : rem;
}
__device__ __forceinline__ unsigned call_number(unsigned v) {
  unsigned c;
  asm volatile("s_waitcnt vmcnt(0)\n\tv_readfirstlane_b32 %0, %1" : "=s"(c) : "v"(v) : "memory");
  return c + 1u;
}

#define DIST 4
#define MAIN_LOOP_ASM \
  "s_getpc_b64 s[72:73]\n" \
  "Lpc_%=:\n" \
  "s_add_u32 s50, s72, Ltk_%=-Lpc_%=\n" \
  "s_addc_u32 s51, s73, 0\n" \
  "s_add_u32 s72, s72, Lwait_%=-Lpc_%=\n" \
  "s_addc_u32 s73, s73, 0\n" \
  "s_mov_b32 s63, 0\n" \
  "s_mov_b32 s66, 0\n" \
  "s_mov_b32 s60, 4\n" \
  "v_mov_b32 v127, 16\n" \
  "v_mov_b32 v112, %[b0]\n" \
  "v_xor_b32 v113, 32, %[b0]\n" \
  "v_xor_b32 v114, 64, %[b0]\n" \
  "v_xor_b32 v115, 96, %[b0]\n" \
  "v_add_u32 v116, 0x10000, v112\n" \
  "v_add_u32 v117, 0x10000, v113\n" \
  "v_add_u32 v118, 0x10000, v114\n" \
  "v_add_u32 v119, 0x10000, v115\n" \
  "v_xor_b32 v192, 16, %[la]\n" \
  "v_xor_b32 v193, 64, %[la]\n" \
  "v_xor_b32 v194, 0x50, %[la]\n" \
  "ds_read_b128 v[160:163], %[la] offset:0\n" \
  "ds_read_b128 v[164:167], v192 offset:0\n" \
  "ds_read_b128 v[168:171], v193 offset:0\n" \
  "ds_read_b128 v[172:175], v194 offset:0\n" \
  "ds_read_b128 v[176:179], %[la] offset:4096\n" \
  "ds_read_b128 v[180:183], v192 offset:4096\n" \
  "ds_read_b128 v[184:187], v193 offset:4096\n" \
  "ds_read_b128 v[188:191], v194 offset:4096\n" \
  "ds_write_b32 %[dn], v127\n" \
  "s_mov_b32 s60, 4\n" \
  "s_swappc_b64 s[70:71], s[72:73]\n" \
  "ds_read_b64_tr_b8 v[192:193], v112 offset:16384\n" \
  "ds_read_b64_tr_b8 v[194:195], v112 offset:17408\n" \
  "ds_read_b64_tr_b8 v[196:197], v112 offset:18432\n" \
  "ds_read_b64_tr_b8 v[198:199], v112 offset:19456\n" \
  "ds_read_b64_tr_b8 v[200:201], v113 offset:16384\n" \
  "ds_read_b64_tr_b8 v[202:203], v113 offset:17408\n" \
  "ds_read_b64_tr_b8 v[204:205], v113 offset:18432\n" \
  "ds_read_b64_tr_b8 v[206:207], v113 offset:19456\n" \
  "ds_read_b64_tr_b8 v[208:209], v114 offset:16384\n" \
  "ds_read_b64_tr_b8 v[210:211], v114 offset:17408\n" \
  "ds_read_b64_tr_b8 v[212:213], v114 offset:18432\n" \
  "ds_read_b64_tr_b8 v[214:215], v114 offset:19456\n" \
  "ds_read_b64_tr_b8 v[216:217], v115 offset:16384\n" \
  "ds_read_b64_tr_b8 v[218:219], v115 offset:17408\n" \
  "ds_read_b64_tr_b8 v[220:221], v115 offset:18432\n" \
  "ds_read_b64_tr_b8 v[222:223], v115 offset:19456\n" \
  "v_add_u32 v127, 1, v127\n" \
  "s_waitcnt lgkmcnt(0)\n" \
  "s_branch Lhalf1_%=\n" \
  "Lloop_%=:\n" \
  "s_waitcnt lgkmcnt(9)\n" \
  "v_mfma_f32_32x32x64_f8f6f4 v[48:63], v[128:135], v[192:199], v[48:63]\n" \
  "ds_read_b96 v[120:122], %[sy]\n" \
  "v_readlane_b32 s62, %[tab], s60\n" \
  "v_xor_b32 v116, 0x8000, v116\n" \
  "v_xor_b32 v117, 0x8000, v117\n" \
  "v_xor_b32 v118, 0x8000, v118\n" \
  "v_xor_b32 v119, 0x8000, v119\n" \
  "ds_read_b64_tr_b8 v[224:225], v116 offset:0\n" \
  "ds_read_b64_tr_b8 v[226:227], v116 offset:1024\n" \
  "ds_read_b64_tr_b8 v[228:229], v116 offset:2048\n" \
  "ds_read_b64_tr_b8 v[230:231], v116 offset:3072\n" \
  "s_add_u32 s60, s60, 1\n" \
  "v_mfma_f32_32x32x64_f8f6f4 v[64:79], v[128:135], v[200:207], v[64:79]\n" \
  "ds_read_b64_tr_b8 v[232:233], v117 offset:0\n" \
  "ds_read_b64_tr_b8 v[234:235], v117 offset:1024\n" \
  "ds_read_b64_tr_b8 v[236:237], v117 offset:2048\n" \
  "ds_read_b64_tr_b8 v[238:239], v117 offset:3072\n" \
  "buffer_load_dwordx4 v[160:163], %[ao], %[rs], s62 offen sc0 sc1\n" \
  "s_waitcnt lgkmcnt(8)\n" \
  "v_mfma_f32_32x32x64_f8f6f4 v[80:95], v[128:135], v[208:215], v[80:95]\n" \
  "v_min3_u32 v120, v120, v121, v122\n" \
  "ds_read_b64_tr_b8 v[240:241], v118 offset:0\n" \
  "ds_read_b64_tr_b8 v[242:243], v118 offset:1024\n" \
  "ds_read_b64_tr_b8 v[244:245], v118 offset:2048\n" \
  "ds_read_b64_tr_b8 v[246:247], v118 offset:3072\n" \
  "buffer_load_dwordx4 v[164:167], %[ao], %[rs], s62 offen offset:1024 sc0 sc1\n" \
  "v_readfirstlane_b32 s61, v120\n" \
  "v_mfma_f32_32x32x64_f8f6f4 v[96:111], v[128:135], v[216:223], v[96:111]\n" \
  "ds_read_b64_tr_b8 v[248:249], v119 offset:0\n" \
  "ds_read_b64_tr_b8 v[250:251], v119 offset:1024\n" \
  "ds_read_b64_tr_b8 v[252:253], v119 offset:2048\n" \
  "ds_read_b64_tr_b8 v[254:255], v119 offset:3072\n" \
  "s_waitcnt vmcnt(6)\n" \
  "v_min3_u32 v126, v136, v137, v138\n" \
  "v_min3_u32 v125, v139, v140, v141\n" \
  "v_min3_u32 v126, v126, v142, v143\n" \
  "v_min_u32 v126, v126, v125\n" \
  "v_cmp_eq_u32 vcc, 0, v126\n" \
  "s_cmp_ge_i32 s60, s61\n" \
  "s_cbranch_vccnz Lsl1_%=\n" \
  "s_cbranch_scc1 Lg0_%=\n" \
  "Lgr0_%=:\n" \
  "s_waitcnt lgkmcnt(8)\n" \
  "v_mfma_f32_32x32x64_f8f6f4 v[48:63], v[136:143], v[224:231], v[48:63]\n" \
  "ds_read_b96 v[120:122], %[sy]\n" \
  "v_readlane_b32 s62, %[tab], s60\n" \
  "ds_read_b64_tr_b8 v[192:193], v112 offset:8192\n" \
  "ds_read_b64_tr_b8 v[194:195], v112 offset:9216\n" \
  "ds_read_b64_tr_b8 v[196:197], v112 offset:10240\n" \
  "ds_read_b64_tr_b8 v[198:199], v112 offset:11264\n" \
  "s_add_u32 s60, s60, 1\n" \
  "v_mfma_f32_32x32x64_f8f6f4 v[64:79], v[136:143], v[232:239], v[64:79]\n" \
  "ds_read_b64_tr_b8 v[200:201], v113 offset:8192\n" \
  "ds_read_b64_tr_b8 v[202:203], v113 offset:9216\n" \
  "ds_read_b64_tr_b8 v[204:205], v113 offset:10240\n" \
  "ds_read_b64_tr_b8 v[206:207], v113 offset:11264\n" \
  "buffer_load_dwordx4 v[168:171], %[ao], %[rs], s62 offen sc0 sc1\n" \
  "s_waitcnt lgkmcnt(8)\n" \
  "v_mfma_f32_32x32x64_f8f6f4 v[80:95], v[136:143], v[240:247], v[80:95]\n" \
  "v_min3_u32 v120, v120, v121, v122\n" \
  "ds_read_b64_tr_b8 v[208:209], v114 offset:8192\n" \
  "ds_read_b64_tr_b8 v[210:211], v114 offset:9216\n" \
  "ds_read_b64_tr_b8 v[212:213], v114 offset:10240\n" \
  "ds_read_b64_tr_b8 v[214:215], v114 offset:11264\n" \
  "buffer_load_dwordx4 v[172:175], %[ao], %[rs], s62 offen offset:1024 sc0 sc1\n" \
  "v_readfirstlane_b32 s61, v120\n" \
  "v_mfma_f32_32x32x64_f8f6f4 v[96:111], v[136:143], v[248:255], v[96:111]\n" \
  "ds_read_b64_tr_b8 v[216:217], v115 offset:8192\n" \
  "ds_read_b64_tr_b8 v[218:219], v115 offset:9216\n" \
  "ds_read_b64_tr_b8 v[220:221], v115 offset:10240\n" \
  "ds_read_b64_tr_b8 v[222:223], v115 offset:11264\n" \
  "v_add_u32 v127, 2, v127\n" \
  "ds_write_b32 %[dn], v127\n" \
  "s_waitcnt vmcnt(6)\n" \
  "v_min3_u32 v126, v144, v145, v146\n" \
  "v_min3_u32 v125, v147, v148, v149\n" \
  "v_min3_u32 v126, v126, v150, v151\n" \
  "v_min_u32 v126, v126, v125\n" \
  "v_cmp_eq_u32 vcc, 0, v126\n" \
  "s_cmp_ge_i32 s60, s61\n" \
  "s_cbranch_vccnz Lsl2_%=\n" \
  "s_cbranch_scc1 Lg1_%=\n" \
  "Lgr1_%=:\n" \
  "s_waitcnt lgkmcnt(9)\n" \
  "v_mfma_f32_32x32x64_f8f6f4 v[48:63], v[144:151], v[192:199], v[48:63]\n" \
  "ds_read_b96 v[120:122], %[sy]\n" \
  "v_readlane_b32 s62, %[tab], s60\n" \
  "ds_read_b64_tr_b8 v[224:225], v116 offset:8192\n" \
  "ds_read_b64_tr_b8 v[226:227], v116 offset:9216\n" \
  "ds_read_b64_tr_b8 v[228:229], v116 offset:10240\n" \
  "ds_read_b64_tr_b8 v[230:231], v116 offset:11264\n" \
  "s_add_u32 s60, s60, 1\n" \
  "v_mfma_f32_32x32x64_f8f6f4 v[64:79], v[144:151], v[200:207], v[64:79]\n" \
  "ds_read_b64_tr_b8 v[232:233], v117 offset:8192\n" \
  "ds_read_b64_tr_b8 v[234:235], v117 offset:9216\n" \
  "ds_read_b64_tr_b8 v[236:237], v117 offset:10240\n" \
  "ds_read_b64_tr_b8 v[238:239], v117 offset:11264\n" \
  "buffer_load_dwordx4 v[176:179], %[ao], %[rs], s62 offen sc0 sc1\n" \
  "s_waitcnt lgkmcnt(8)\n" \
  "v_mfma_f32_32x32x64_f8f6f4 v[80:95], v[144:151], v[208:215], v[80:95]\n" \
  "v_min3_u32 v120, v120, v121, v122\n" \
  "ds_read_b64_tr_b8 v[240:241], v118 offset:8192\n" \
  "ds_read_b64_tr_b8 v[242:243], v118 offset:9216\n" \
  "ds_read_b64_tr_b8 v[244:245], v118 offset:10240\n" \
  "ds_read_b64_tr_b8 v[246:247], v118 offset:11264\n" \
  "buffer_load_dwordx4 v[180:183], %[ao], %[rs], s62 offen offset:1024 sc0 sc1\n" \
  "v_readfirstlane_b32 s61, v120\n" \
  "v_mfma_f32_32x32x64_f8f6f4 v[96:111], v[144:151], v[216:223], v[96:111]\n" \
  "ds_read_b64_tr_b8 v[248:249], v119 offset:8192\n" \
  "ds_read_b64_tr_b8 v[250:251], v119 offset:9216\n" \
  "ds_read_b64_tr_b8 v[252:253], v119 offset:10240\n" \
  "ds_read_b64_tr_b8 v[254:255], v119 offset:11264\n" \
  "s_waitcnt vmcnt(6)\n" \
  "v_min3_u32 v126, v152, v153, v154\n" \
  "v_min3_u32 v125, v155, v156, v157\n" \
  "v_min3_u32 v126, v126, v158, v159\n" \
  "v_min_u32 v126, v126, v125\n" \
  "v_cmp_eq_u32 vcc, 0, v126\n" \
  "s_cmp_ge_i32 s60, s61\n" \
  "s_cbranch_vccnz Lsl3_%=\n" \
  "s_cbranch_scc1 Lg2_%=\n" \
  "Lgr2_%=:\n" \
  "s_waitcnt lgkmcnt(8)\n" \
  "v_mfma_f32_32x32x64_f8f6f4 v[48:63], v[152:159], v[224:231], v[48:63]\n" \
  "ds_read_b96 v[120:122], %[sy]\n" \
  "v_readlane_b32 s62, %[tab], s60\n" \
  "ds_read_b64_tr_b8 v[192:193], v112 offset:16384\n" \
  "ds_read_b64_tr_b8 v[194:195], v112 offset:17408\n" \
  "ds_read_b64_tr_b8 v[196:197], v112 offset:18432\n" \
  "ds_read_b64_tr_b8 v[198:199], v112 offset:19456\n" \
  "s_add_u32 s60, s60, 1\n" \
  "v_mfma_f32_32x32x64_f8f6f4 v[64:79], v[152:159], v[232:239], v[64:79]\n" \
  "ds_read_b64_tr_b8 v[200:201], v113 offset:16384\n" \
  "ds_read_b64_tr_b8 v[202:203], v113 offset:17408\n" \
  "ds_read_b64_tr_b8 v[204:205], v113 offset:18432\n" \
  "ds_read_b64_tr_b8 v[206:207], v113 offset:19456\n" \
  "buffer_load_dwordx4 v[184:187], %[ao], %[rs], s62 offen sc0 sc1\n" \
  "s_waitcnt lgkmcnt(8)\n" \
  "v_mfma_f32_32x32x64_f8f6f4 v[80:95], v[152:159], v[240:247], v[80:95]\n" \
  "v_min3_u32 v120, v120, v121, v122\n" \
  "ds_read_b64_tr_b8 v[208:209], v114 offset:16384\n" \
  "ds_read_b64_tr_b8 v[210:211], v114 offset:17408\n" \
  "ds_read_b64_tr_b8 v[212:213], v114 offset:18432\n" \
  "ds_read_b64_tr_b8 v[214:215], v114 offset:19456\n" \
  "buffer_load_dwordx4 v[188:191], %[ao], %[rs], s62 offen offset:1024 sc0 sc1\n" \
  "v_readfirstlane_b32 s61, v120\n" \
  "v_mfma_f32_32x32x64_f8f6f4 v[96:111], v[152:159], v[248:255], v[96:111]\n" \
  "ds_read_b64_tr_b8 v[216:217], v115 offset:16384\n" \
  "ds_read_b64_tr_b8 v[218:219], v115 offset:17408\n" \
  "ds_read_b64_tr_b8 v[220:221], v115 offset:18432\n" \
  "ds_read_b64_tr_b8 v[222:223], v115 offset:19456\n" \
  "v_add_u32 v127, 2, v127\n" \
  "ds_write_b32 %[dn], v127\n" \
  "s_waitcnt vmcnt(6)\n" \
  "v_min3_u32 v126, v160, v161, v162\n" \
  "v_min3_u32 v125, v163, v164, v165\n" \
  "v_min3_u32 v126, v126, v166, v167\n" \
  "v_min_u32 v126, v126, v125\n" \
  "v_cmp_eq_u32 vcc, 0, v126\n" \
  "s_cmp_ge_i32 s60, s61\n" \
  "s_cbranch_vccnz Lsl4_%=\n" \
  "s_cbranch_scc1 Lg3_%=\n" \
  "Lgr3_%=:\n" \
  "s_cmp_eq_u32 s60, 20\n" \
  "s_cbranch_scc1 Lfin_%=\n" \
  "Lhalf1_%=:\n" \
  "s_waitcnt lgkmcnt(9)\n" \
  "v_mfma_f32_32x32x64_f8f6f4 v[48:63], v[160:167], v[192:199], v[48:63]\n" \
  "ds_read_b96 v[120:122], %[sy]\n" \
  "v_readlane_b32 s62, %[tab], s60\n" \
  "ds_read_b64_tr_b8 v[224:225], v116 offset:16384\n" \
  "ds_read_b64_tr_b8 v[226:227], v116 offset:17408\n" \
  "ds_read_b64_tr_b8 v[228:229], v116 offset:18432\n" \
  "ds_read_b64_tr_b8 v[230:231], v116 offset:19456\n" \
  "s_add_u32 s60, s60, 1\n" \
  "v_mfma_f32_32x32x64_f8f6f4 v[64:79], v[160:167], v[200:207], v[64:79]\n" \
  "ds_read_b64_tr_b8 v[232:233], v117 offset:16384\n" \
  "ds_read_b64_tr_b8 v[234:235], v117 offset:17408\n" \
  "ds_read_b64_tr_b8 v[236:237], v117 offset:18432\n" \
  "ds_read_b64_tr_b8 v[238:239], v117 offset:19456\n" \
  "buffer_load_dwordx4 v[128:131], %[ao], %[rs], s62 offen sc0 sc1\n" \
  "s_waitcnt lgkmcnt(8)\n" \
  "v_mfma_f32_32x32x64_f8f6f4 v[80:95], v[160:167], v[208:215], v[80:95]\n" \
  "v_min3_u32 v120, v120, v121, v122\n" \
  "ds_read_b64_tr_b8 v[240:241], v118 offset:16384\n" \
  "ds_read_b64_tr_b8 v[242:243], v118 offset:17408\n" \
  "ds_read_b64_tr_b8 v[244:245], v118 offset:18432\n" \
  "ds_read_b64_tr_b8 v[246:247], v118 offset:19456\n" \
  "buffer_load_dwordx4 v[132:135], %[ao], %[rs], s62 offen offset:1024 sc0 sc1\n" \
  "v_readfirstlane_b32 s61, v120\n" \
  "v_mfma_f32_32x32x64_f8f6f4 v[96:111], v[160:167], v[216:223], v[96:111]\n" \
  "ds_read_b64_tr_b8 v[248:249], v119 offset:16384\n" \
  "ds_read_b64_tr_b8 v[250:251], v119 offset:17408\n" \
  "ds_read_b64_tr_b8 v[252:253], v119 offset:18432\n" \
  "ds_read_b64_tr_b8 v[254:255], v119 offset:19456\n" \
  "s_waitcnt vmcnt(6)\n" \
  "v_min3_u32 v126, v168, v169, v170\n" \
  "v_min3_u32 v125, v171, v172, v173\n" \
  "v_min3_u32 v126, v126, v174, v175\n" \
  "v_min_u32 v126, v126, v125\n" \
  "v_cmp_eq_u32 vcc, 0, v126\n" \
  "s_cmp_ge_i32 s60, s61\n" \
  "s_cbranch_vccnz Lsl5_%=\n" \
  "s_cbranch_scc1 Lg4_%=\n" \
  "Lgr4_%=:\n" \
  "s_waitcnt lgkmcnt(8)\n" \
  "v_mfma_f32_32x32x64_f8f6f4 v[48:63], v[168:175], v[224:231], v[48:63]\n" \
  "ds_read_b96 v[120:122], %[sy]\n" \
  "v_readlane_b32 s62, %[tab], s60\n" \
  "ds_read_b64_tr_b8 v[192:193], v112 offset:24576\n" \
  "ds_read_b64_tr_b8 v[194:195], v112 offset:25600\n" \
  "ds_read_b64_tr_b8 v[196:197], v112 offset:26624\n" \
  "ds_read_b64_tr_b8 v[198:199], v112 offset:27648\n" \
  "s_add_u32 s60, s60, 1\n" \
  "v_mfma_f32_32x32x64_f8f6f4 v[64:79], v[168:175], v[232:239], v[64:79]\n" \
  "ds_read_b64_tr_b8 v[200:201], v113 offset:24576\n" \
  "ds_read_b64_tr_b8 v[202:203], v113 offset:25600\n" \
  "ds_read_b64_tr_b8 v[204:205], v113 offset:26624\n" \
  "ds_read_b64_tr_b8 v[206:207], v113 offset:27648\n" \
  "buffer_load_dwordx4 v[136:139], %[ao], %[rs], s62 offen sc0 sc1\n" \
  "s_waitcnt lgkmcnt(8)\n" \
  "v_mfma_f32_32x32x64_f8f6f4 v[80:95], v[168:175], v[240:247], v[80:95]\n" \
  "v_min3_u32 v120, v120, v121, v122\n" \
  "ds_read_b64_tr_b8 v[208:209], v114 offset:24576\n" \
  "ds_read_b64_tr_b8 v[210:211], v114 offset:25600\n" \
  "ds_read_b64_tr_b8 v[212:213], v114 offset:26624\n" \
  "ds_read_b64_tr_b8 v[214:215], v114 offset:27648\n" \
  "buffer_load_dwordx4 v[140:143], %[ao], %[rs], s62 offen offset:1024 sc0 sc1\n" \
  "v_readfirstlane_b32 s61, v120\n" \
  "v_mfma_f32_32x32x64_f8f6f4 v[96:111], v[168:175], v[248:255], v[96:111]\n" \
  "ds_read_b64_tr_b8 v[216:217], v115 offset:24576\n" \
  "ds_read_b64_tr_b8 v[218:219], v115 offset:25600\n" \
  "ds_read_b64_tr_b8 v[220:221], v115 offset:26624\n" \
  "ds_read_b64_tr_b8 v[222:223], v115 offset:27648\n" \
  "v_add_u32 v127, 2, v127\n" \
  "ds_write_b32 %[dn], v127\n" \
  "s_waitcnt vmcnt(6)\n" \
  "v_min3_u32 v126, v176, v177, v178\n" \
  "v_min3_u32 v125, v179, v180, v181\n" \
  "v_min3_u32 v126, v126, v182, v183\n" \
  "v_min_u32 v126, v126, v125\n" \
  "v_cmp_eq_u32 vcc, 0, v126\n" \
  "s_cmp_ge_i32 s60, s61\n" \
  "s_cbranch_vccnz Lsl6_%=\n" \
  "s_cbranch_scc1 Lg5_%=\n" \
  "Lgr5_%=:\n" \
  "s_waitcnt lgkmcnt(9)\n" \
  "v_mfma_f32_32x32x64_f8f6f4 v[48:63], v[176:183], v[192:199], v[48:63]\n" \
  "ds_read_b96 v[120:122], %[sy]\n" \
  "v_readlane_b32 s62, %[tab], s60\n" \
  "ds_read_b64_tr_b8 v[224:225], v116 offset:24576\n" \
  "ds_read_b64_tr_b8 v[226:227], v116 offset:25600\n" \
  "ds_read_b64_tr_b8 v[228:229], v116 offset:26624\n" \
  "ds_read_b64_tr_b8 v[230:231], v116 offset:27648\n" \
  "s_add_u32 s60, s60, 1\n" \
  "v_mfma_f32_32x32x64_f8f6f4 v[64:79], v[176:183], v[200:207], v[64:79]\n" \
  "ds_read_b64_tr_b8 v[232:233], v117 offset:24576\n" \
  "ds_read_b64_tr_b8 v[234:235], v117 offset:25600\n" \
  "ds_read_b64_tr_b8 v[236:237], v117 offset:26624\n" \
  "ds_read_b64_tr_b8 v[238:239], v117 offset:27648\n" \
  "buffer_load_dwordx4 v[144:147], %[ao], %[rs], s62 offen sc0 sc1\n" \
  "s_waitcnt lgkmcnt(8)\n" \
  "v_mfma_f32_32x32x64_f8f6f4 v[80:95], v[176:183], v[208:215], v[80:95]\n" \
  "v_min3_u32 v120, v120, v121, v122\n" \
  "ds_read_b64_tr_b8 v[240:241], v118 offset:24576\n" \
  "ds_read_b64_tr_b8 v[242:243], v118 offset:25600\n" \
  "ds_read_b64_tr_b8 v[244:245], v118 offset:26624\n" \
  "ds_read_b64_tr_b8 v[246:247], v118 offset:27648\n" \
  "buffer_load_dwordx4 v[148:151], %[ao], %[rs], s62 offen offset:1024 sc0 sc1\n" \
  "v_readfirstlane_b32 s61, v120\n" \
  "v_mfma_f32_32x32x64_f8f6f4 v[96:111], v[176:183], v[216:223], v[96:111]\n" \
  "ds_read_b64_tr_b8 v[248:249], v119 offset:24576\n" \
  "ds_read_b64_tr_b8 v[250:251], v119 offset:25600\n" \
  "ds_read_b64_tr_b8 v[252:253], v119 offset:26624\n" \
  "ds_read_b64_tr_b8 v[254:255], v119 offset:27648\n" \
  "s_waitcnt vmcnt(6)\n" \
  "v_min3_u32 v126, v184, v185, v186\n" \
  "v_min3_u32 v125, v187, v188, v189\n" \
  "v_min3_u32 v126, v126, v190, v191\n" \
  "v_min_u32 v126, v126, v125\n" \
  "v_cmp_eq_u32 vcc, 0, v126\n" \
  "s_cmp_ge_i32 s60, s61\n" \
  "s_cbranch_vccnz Lsl7_%=\n" \
  "s_cbranch_scc1 Lg6_%=\n" \
  "Lgr6_%=:\n" \
  "s_waitcnt lgkmcnt(8)\n" \
  "v_mfma_f32_32x32x64_f8f6f4 v[48:63], v[184:191], v[224:231], v[48:63]\n" \
  "ds_read_b96 v[120:122], %[sy]\n" \
  "v_readlane_b32 s62, %[tab], s60\n" \
  "v_xor_b32 v112, 0x8000, v112\n" \
  "v_xor_b32 v113, 0x8000, v113\n" \
  "v_xor_b32 v114, 0x8000, v114\n" \
  "v_xor_b32 v115, 0x8000, v115\n" \
  "ds_read_b64_tr_b8 v[192:193], v112 offset:0\n" \
  "ds_read_b64_tr_b8 v[194:195], v112 offset:1024\n" \
  "ds_read_b64_tr_b8 v[196:197], v112 offset:2048\n" \
  "ds_read_b64_tr_b8 v[198:199], v112 offset:3072\n" \
  "s_add_u32 s60, s60, 1\n" \
  "v_mfma_f32_32x32x64_f8f6f4 v[64:79], v[184:191], v[232:239], v[64:79]\n" \
  "ds_read_b64_tr_b8 v[200:201], v113 offset:0\n" \
  "ds_read_b64_tr_b8 v[202:203], v113 offset:1024\n" \
  "ds_read_b64_tr_b8 v[204:205], v113 offset:2048\n" \
  "ds_read_b64_tr_b8 v[206:207], v113 offset:3072\n" \
  "buffer_load_dwordx4 v[152:155], %[ao], %[rs], s62 offen sc0 sc1\n" \
  "s_waitcnt lgkmcnt(8)\n" \
  "v_mfma_f32_32x32x64_f8f6f4 v[80:95], v[184:191], v[240:247], v[80:95]\n" \
  "v_min3_u32 v120, v120, v121, v122\n" \
  "ds_read_b64_tr_b8 v[208:209], v114 offset:0\n" \
  "ds_read_b64_tr_b8 v[210:211], v114 offset:1024\n" \
  "ds_read_b64_tr_b8 v[212:213], v114 offset:2048\n" \
  "ds_read_b64_tr_b8 v[214:215], v114 offset:3072\n" \
  "buffer_load_dwordx4 v[156:159], %[ao], %[rs], s62 offen offset:1024 sc0 sc1\n" \
  "v_readfirstlane_b32 s61, v120\n" \
  "v_mfma_f32_32x32x64_f8f6f4 v[96:111], v[184:191], v[248:255], v[96:111]\n" \
  "ds_read_b64_tr_b8 v[216:217], v115 offset:0\n" \
  "ds_read_b64_tr_b8 v[218:219], v115 offset:1024\n" \
  "ds_read_b64_tr_b8 v[220:221], v115 offset:2048\n" \
  "ds_read_b64_tr_b8 v[222:223], v115 offset:3072\n" \
  "v_add_u32 v127, 2, v127\n" \
  "ds_write_b32 %[dn], v127\n" \
  "s_waitcnt vmcnt(6)\n" \
  "v_min3_u32 v126, v128, v129, v130\n" \
  "v_min3_u32 v125, v131, v132, v133\n" \
  "v_min3_u32 v126, v126, v134, v135\n" \
  "v_min_u32 v126, v126, v125\n" \
  "v_cmp_eq_u32 vcc, 0, v126\n" \
  "s_cmp_ge_i32 s60, s61\n" \
  "s_cbranch_vccnz Lsl0_%=\n" \
  "s_cbranch_scc1 Lg7_%=\n" \
  "Lgr7_%=:\n" \
  "s_branch Lloop_%=\n" \
  "Lg0_%=:\n" \
  "s_swappc_b64 s[70:71], s[72:73]\n" \
  "s_branch Lgr0_%=\n" \
  "Lg1_%=:\n" \
  "s_swappc_b64 s[70:71], s[72:73]\n" \
  "s_branch Lgr1_%=\n" \
  "Lg2_%=:\n" \
  "s_swappc_b64 s[70:71], s[72:73]\n" \
  "s_branch Lgr2_%=\n" \
  "Lg3_%=:\n" \
  "s_swappc_b64 s[70:71], s[72:73]\n" \
  "s_branch Lgr3_%=\n" \
  "Lg4_%=:\n" \
  "s_swappc_b64 s[70:71], s[72:73]\n" \
  "s_branch Lgr4_%=\n" \
  "Lg5_%=:\n" \
  "s_swappc_b64 s[70:71], s[72:73]\n" \
  "s_branch Lgr5_%=\n" \
  "Lg6_%=:\n" \
  "s_swappc_b64 s[70:71], s[72:73]\n" \
  "s_branch Lgr6_%=\n" \
  "Lg7_%=:\n" \
  "s_swappc_b64 s[70:71], s[72:73]\n" \
  "s_branch Lgr7_%=\n" \
  "Lsl0_%=:\n" \
  "s_sub_u32 s64, s60, 4\n" \
  "v_readlane_b32 s52, %[tab], s64\n" \
  "s_add_u32 s64, s64, 1\n" \
  "v_readlane_b32 s53, %[tab], s64\n" \
  "s_add_u32 s64, s64, 1\n" \
  "v_readlane_b32 s54, %[tab], s64\n" \
  "s_add_u32 s64, s64, 1\n" \
  "v_readlane_b32 s55, %[tab], s64\n" \
  "s_add_u32 s64, s64, 1\n" \
  "Lsll0_%=:\n" \
  "s_add_u32 s66, s66, 1\n" \
  "s_and_b32 s64, s66, 0x3f\n" \
  "s_cbranch_scc1 Lsk0_%=\n" \
  "s_swappc_b64 s[70:71], s[50:51]\n" \
  "Lsk0_%=:\n" \
  "buffer_load_dwordx4 v[128:131], %[ao], %[rs], s52 offen sc0 sc1\n" \
  "buffer_load_dwordx4 v[132:135], %[ao], %[rs], s52 offen offset:1024 sc0 sc1\n" \
  "buffer_load_dwordx4 v[136:139], %[ao], %[rs], s53 offen sc0 sc1\n" \
  "buffer_load_dwordx4 v[140:143], %[ao], %[rs], s53 offen offset:1024 sc0 sc1\n" \
  "buffer_load_dwordx4 v[144:147], %[ao], %[rs], s54 offen sc0 sc1\n" \
  "buffer_load_dwordx4 v[148:151], %[ao], %[rs], s54 offen offset:1024 sc0 sc1\n" \
  "buffer_load_dwordx4 v[152:155], %[ao], %[rs], s55 offen sc0 sc1\n" \
  "buffer_load_dwordx4 v[156:159], %[ao], %[rs], s55 offen offset:1024 sc0 sc1\n" \
  "s_waitcnt vmcnt(6)\n" \
  "v_min3_u32 v126, v128, v129, v130\n" \
  "v_min3_u32 v125, v131, v132, v133\n" \
  "v_min3_u32 v126, v126, v134, v135\n" \
  "v_min_u32 v126, v126, v125\n" \
  "v_cmp_eq_u32 vcc, 0, v126\n" \
  "s_cbranch_vccz Lg7_%=\n" \
  "s_sleep 2\n" \
  "s_branch Lsll0_%=\n" \
  "Lsl1_%=:\n" \
  "s_sub_u32 s64, s60, 4\n" \
  "v_readlane_b32 s52, %[tab], s64\n" \
  "s_add_u32 s64, s64, 1\n" \
  "v_readlane_b32 s53, %[tab], s64\n" \
  "s_add_u32 s64, s64, 1\n" \
  "v_readlane_b32 s54, %[tab], s64\n" \
  "s_add_u32 s64, s64, 1\n" \
  "v_readlane_b32 s55, %[tab], s64\n" \
  "s_add_u32 s64, s64, 1\n" \
  "Lsll1_%=:\n" \
  "s_add_u32 s66, s66, 1\n" \
  "s_and_b32 s64, s66, 0x3f\n" \
  "s_cbranch_scc1 Lsk1_%=\n" \
  "s_swappc_b64 s[70:71], s[50:51]\n" \
  "Lsk1_%=:\n" \
  "buffer_load_dwordx4 v[136:139], %[ao], %[rs], s52 offen sc0 sc1\n" \
  "buffer_load_dwordx4 v[140:143], %[ao], %[rs], s52 offen offset:1024 sc0 sc1\n" \
  "buffer_load_dwordx4 v[144:147], %[ao], %[rs], s53 offen sc0 sc1\n" \
  "buffer_load_dwordx4 v[148:151], %[ao], %[rs], s53 offen offset:1024 sc0 sc1\n" \
  "buffer_load_dwordx4 v[152:155], %[ao], %[rs], s54 offen sc0 sc1\n" \
  "buffer_load_dwordx4 v[156:159], %[ao], %[rs], s54 offen offset:1024 sc0 sc1\n" \
  "buffer_load_dwordx4 v[160:163], %[ao], %[rs], s55 offen sc0 sc1\n" \
  "buffer_load_dwordx4 v[164:167], %[ao], %[rs], s55 offen offset:1024 sc0 sc1\n" \
  "s_waitcnt vmcnt(6)\n" \
  "v_min3_u32 v126, v136, v137, v138\n" \
  "v_min3_u32 v125, v139, v140, v141\n" \
  "v_min3_u32 v126, v126, v142, v143\n" \
  "v_min_u32 v126, v126, v125\n" \
  "v_cmp_eq_u32 vcc, 0, v126\n" \
  "s_cbranch_vccz Lg0_%=\n" \
  "s_sleep 2\n" \
  "s_branch Lsll1_%=\n" \
  "Lsl2_%=:\n" \
  "s_sub_u32 s64, s60, 4\n" \
  "v_readlane_b32 s52, %[tab], s64\n" \
  "s_add_u32 s64, s64, 1\n" \
  "v_readlane_b32 s53, %[tab], s64\n" \
  "s_add_u32 s64, s64, 1\n" \
  "v_readlane_b32 s54, %[tab], s64\n" \
  "s_add_u32 s64, s64, 1\n" \
  "v_readlane_b32 s55, %[tab], s64\n" \
  "s_add_u32 s64, s64, 1\n" \
  "Lsll2_%=:\n" \
  "s_add_u32 s66, s66, 1\n" \
  "s_and_b32 s64, s66, 0x3f\n" \
  "s_cbranch_scc1 Lsk2_%=\n" \
  "s_swappc_b64 s[70:71], s[50:51]\n" \
  "Lsk2_%=:\n" \
  "buffer_load_dwordx4 v[144:147], %[ao], %[rs], s52 offen sc0 sc1\n" \
  "buffer_load_dwordx4 v[148:151], %[ao], %[rs], s52 offen offset:1024 sc0 sc1\n" \
  "buffer_load_dwordx4 v[152:155], %[ao], %[rs], s53 offen sc0 sc1\n" \
  "buffer_load_dwordx4 v[156:159], %[ao], %[rs], s53 offen offset:1024 sc0 sc1\n" \
  "buffer_load_dwordx4 v[160:163], %[ao], %[rs], s54 offen sc0 sc1\n" \
  "buffer_load_dwordx4 v[164:167], %[ao], %[rs], s54 offen offset:1024 sc0 sc1\n" \
  "buffer_load_dwordx4 v[168:171], %[ao], %[rs], s55 offen sc0 sc1\n" \
  "buffer_load_dwordx4 v[172:175], %[ao], %[rs], s55 offen offset:1024 sc0 sc1\n" \
  "s_waitcnt vmcnt(6)\n" \
  "v_min3_u32 v126, v144, v145, v146\n" \
  "v_min3_u32 v125, v147, v148, v149\n" \
  "v_min3_u32 v126, v126, v150, v151\n" \
  "v_min_u32 v126, v126, v125\n" \
  "v_cmp_eq_u32 vcc, 0, v126\n" \
  "s_cbranch_vccz Lg1_%=\n" \
  "s_sleep 2\n" \
  "s_branch Lsll2_%=\n" \
  "Lsl3_%=:\n" \
  "s_sub_u32 s64, s60, 4\n" \
  "v_readlane_b32 s52, %[tab], s64\n" \
  "s_add_u32 s64, s64, 1\n" \
  "v_readlane_b32 s53, %[tab], s64\n" \
  "s_add_u32 s64, s64, 1\n" \
  "v_readlane_b32 s54, %[tab], s64\n" \
  "s_add_u32 s64, s64, 1\n" \
  "v_readlane_b32 s55, %[tab], s64\n" \
  "s_add_u32 s64, s64, 1\n" \
  "Lsll3_%=:\n" \
  "s_add_u32 s66, s66, 1\n" \
  "s_and_b32 s64, s66, 0x3f\n" \
  "s_cbranch_scc1 Lsk3_%=\n" \
  "s_swappc_b64 s[70:71], s[50:51]\n" \
  "Lsk3_%=:\n" \
  "buffer_load_dwordx4 v[152:155], %[ao], %[rs], s52 offen sc0 sc1\n" \
  "buffer_load_dwordx4 v[156:159], %[ao], %[rs], s52 offen offset:1024 sc0 sc1\n" \
  "buffer_load_dwordx4 v[160:163], %[ao], %[rs], s53 offen sc0 sc1\n" \
  "buffer_load_dwordx4 v[164:167], %[ao], %[rs], s53 offen offset:1024 sc0 sc1\n" \
  "buffer_load_dwordx4 v[168:171], %[ao], %[rs], s54 offen sc0 sc1\n" \
  "buffer_load_dwordx4 v[172:175], %[ao], %[rs], s54 offen offset:1024 sc0 sc1\n" \
  "buffer_load_dwordx4 v[176:179], %[ao], %[rs], s55 offen sc0 sc1\n" \
  "buffer_load_dwordx4 v[180:183], %[ao], %[rs], s55 offen offset:1024 sc0 sc1\n" \
  "s_waitcnt vmcnt(6)\n" \
  "v_min3_u32 v126, v152, v153, v154\n" \
  "v_min3_u32 v125, v155, v156, v157\n" \
  "v_min3_u32 v126, v126, v158, v159\n" \
  "v_min_u32 v126, v126, v125\n" \
  "v_cmp_eq_u32 vcc, 0, v126\n" \
  "s_cbranch_vccz Lg2_%=\n" \
  "s_sleep 2\n" \
  "s_branch Lsll3_%=\n" \
  "Lsl4_%=:\n" \
  "s_sub_u32 s64, s60, 4\n" \
  "v_readlane_b32 s52, %[tab], s64\n" \
  "s_add_u32 s64, s64, 1\n" \
  "v_readlane_b32 s53, %[tab], s64\n" \
  "s_add_u32 s64, s64, 1\n" \
  "v_readlane_b32 s54, %[tab], s64\n" \
  "s_add_u32 s64, s64, 1\n" \
  "v_readlane_b32 s55, %[tab], s64\n" \
  "s_add_u32 s64, s64, 1\n" \
  "Lsll4_%=:\n" \
  "s_add_u32 s66, s66, 1\n" \
  "s_and_b32 s64, s66, 0x3f\n" \
  "s_cbranch_scc1 Lsk4_%=\n" \
  "s_swappc_b64 s[70:71], s[50:51]\n" \
  "Lsk4_%=:\n" \
  "buffer_load_dwordx4 v[160:163], %[ao], %[rs], s52 offen sc0 sc1\n" \
  "buffer_load_dwordx4 v[164:167], %[ao], %[rs], s52 offen offset:1024 sc0 sc1\n" \
  "buffer_load_dwordx4 v[168:171], %[ao], %[rs], s53 offen sc0 sc1\n" \
  "buffer_load_dwordx4 v[172:175], %[ao], %[rs], s53 offen offset:1024 sc0 sc1\n" \
  "buffer_load_dwordx4 v[176:179], %[ao], %[rs], s54 offen sc0 sc1\n" \
  "buffer_load_dwordx4 v[180:183], %[ao], %[rs], s54 offen offset:1024 sc0 sc1\n" \
  "buffer_load_dwordx4 v[184:187], %[ao], %[rs], s55 offen sc0 sc1\n" \
  "buffer_load_dwordx4 v[188:191], %[ao], %[rs], s55 offen offset:1024 sc0 sc1\n" \
  "s_waitcnt vmcnt(6)\n" \
  "v_min3_u32 v126, v160, v161, v162\n" \
  "v_min3_u32 v125, v163, v164, v165\n" \
  "v_min3_u32 v126, v126, v166, v167\n" \
  "v_min_u32 v126, v126, v125\n" \
  "v_cmp_eq_u32 vcc, 0, v126\n" \
  "s_cbranch_vccz Lg3_%=\n" \
  "s_sleep 2\n" \
  "s_branch Lsll4_%=\n" \
  "Lsl5_%=:\n" \
  "s_cmp_lt_u32 s60, 8\n" \
  "s_cbranch_scc1 Lg4_%=\n" \
  "s_sub_u32 s64, s60, 4\n" \
  "v_readlane_b32 s52, %[tab], s64\n" \
  "s_add_u32 s64, s64, 1\n" \
  "v_readlane_b32 s53, %[tab], s64\n" \
  "s_add_u32 s64, s64, 1\n" \
  "v_readlane_b32 s54, %[tab], s64\n" \
  "s_add_u32 s64, s64, 1\n" \
  "v_readlane_b32 s55, %[tab], s64\n" \
  "s_add_u32 s64, s64, 1\n" \
  "Lsll5_%=:\n" \
  "s_add_u32 s66, s66, 1\n" \
  "s_and_b32 s64, s66, 0x3f\n" \
  "s_cbranch_scc1 Lsk5_%=\n" \
  "s_swappc_b64 s[70:71], s[50:51]\n" \
  "Lsk5_%=:\n" \
  "buffer_load_dwordx4 v[168:171], %[ao], %[rs], s52 offen sc0 sc1\n" \
  "buffer_load_dwordx4 v[172:175], %[ao], %[rs], s52 offen offset:1024 sc0 sc1\n" \
  "buffer_load_dwordx4 v[176:179], %[ao], %[rs], s53 offen sc0 sc1\n" \
  "buffer_load_dwordx4 v[180:183], %[ao], %[rs], s53 offen offset:1024 sc0 sc1\n" \
  "buffer_load_dwordx4 v[184:187], %[ao], %[rs], s54 offen sc0 sc1\n" \
  "buffer_load_dwordx4 v[188:191], %[ao], %[rs], s54 offen offset:1024 sc0 sc1\n" \
  "buffer_load_dwordx4 v[128:131], %[ao], %[rs], s55 offen sc0 sc1\n" \
  "buffer_load_dwordx4 v[132:135], %[ao], %[rs], s55 offen offset:1024 sc0 sc1\n" \
  "s_waitcnt vmcnt(6)\n" \
  "v_min3_u32 v126, v168, v169, v170\n" \
  "v_min3_u32 v125, v171, v172, v173\n" \
  "v_min3_u32 v126, v126, v174, v175\n" \
  "v_min_u32 v126, v126, v125\n" \
  "v_cmp_eq_u32 vcc, 0, v126\n" \
  "s_cbranch_vccz Lg4_%=\n" \
  "s_sleep 2\n" \
  "s_branch Lsll5_%=\n" \
  "Lsl6_%=:\n" \
  "s_cmp_lt_u32 s60, 8\n" \
  "s_cbranch_scc1 Lg5_%=\n" \
  "s_sub_u32 s64, s60, 4\n" \
  "v_readlane_b32 s52, %[tab], s64\n" \
  "s_add_u32 s64, s64, 1\n" \
  "v_readlane_b32 s53, %[tab], s64\n" \
  "s_add_u32 s64, s64, 1\n" \
  "v_readlane_b32 s54, %[tab], s64\n" \
  "s_add_u32 s64, s64, 1\n" \
  "v_readlane_b32 s55, %[tab], s64\n" \
  "s_add_u32 s64, s64, 1\n" \
  "Lsll6_%=:\n" \
  "s_add_u32 s66, s66, 1\n" \
  "s_and_b32 s64, s66, 0x3f\n" \
  "s_cbranch_scc1 Lsk6_%=\n" \
  "s_swappc_b64 s[70:71], s[50:51]\n" \
  "Lsk6_%=:\n" \
  "buffer_load_dwordx4 v[176:179], %[ao], %[rs], s52 offen sc0 sc1\n" \
  "buffer_load_dwordx4 v[180:183], %[ao], %[rs], s52 offen offset:1024 sc0 sc1\n" \
  "buffer_load_dwordx4 v[184:187], %[ao], %[rs], s53 offen sc0 sc1\n" \
  "buffer_load_dwordx4 v[188:191], %[ao], %[rs], s53 offen offset:1024 sc0 sc1\n" \
  "buffer_load_dwordx4 v[128:131], %[ao], %[rs], s54 offen sc0 sc1\n" \
  "buffer_load_dwordx4 v[132:135], %[ao], %[rs], s54 offen offset:1024 sc0 sc1\n" \
  "buffer_load_dwordx4 v[136:139], %[ao], %[rs], s55 offen sc0 sc1\n" \
  "buffer_load_dwordx4 v[140:143], %[ao], %[rs], s55 offen offset:1024 sc0 sc1\n" \
  "s_waitcnt vmcnt(6)\n" \
  "v_min3_u32 v126, v176, v177, v178\n" \
  "v_min3_u32 v125, v179, v180, v181\n" \
  "v_min3_u32 v126, v126, v182, v183\n" \
  "v_min_u32 v126, v126, v125\n" \
  "v_cmp_eq_u32 vcc, 0, v126\n" \
  "s_cbranch_vccz Lg5_%=\n" \
  "s_sleep 2\n" \
  "s_branch Lsll6_%=\n" \
  "Lsl7_%=:\n" \
  "s_cmp_lt_u32 s60, 8\n" \
  "s_cbranch_scc1 Lg6_%=\n" \
  "s_sub_u32 s64, s60, 4\n" \
  "v_readlane_b32 s52, %[tab], s64\n" \
  "s_add_u32 s64, s64, 1\n" \
  "v_readlane_b32 s53, %[tab], s64\n" \
  "s_add_u32 s64, s64, 1\n" \
  "v_readlane_b32 s54, %[tab], s64\n" \
  "s_add_u32 s64, s64, 1\n" \
  "v_readlane_b32 s55, %[tab], s64\n" \
  "s_add_u32 s64, s64, 1\n" \
  "Lsll7_%=:\n" \
  "s_add_u32 s66, s66, 1\n" \
  "s_and_b32 s64, s66, 0x3f\n" \
  "s_cbranch_scc1 Lsk7_%=\n" \
  "s_swappc_b64 s[70:71], s[50:51]\n" \
  "Lsk7_%=:\n" \
  "buffer_load_dwordx4 v[184:187], %[ao], %[rs], s52 offen sc0 sc1\n" \
  "buffer_load_dwordx4 v[188:191], %[ao], %[rs], s52 offen offset:1024 sc0 sc1\n" \
  "buffer_load_dwordx4 v[128:131], %[ao], %[rs], s53 offen sc0 sc1\n" \
  "buffer_load_dwordx4 v[132:135], %[ao], %[rs], s53 offen offset:1024 sc0 sc1\n" \
  "buffer_load_dwordx4 v[136:139], %[ao], %[rs], s54 offen sc0 sc1\n" \
  "buffer_load_dwordx4 v[140:143], %[ao], %[rs], s54 offen offset:1024 sc0 sc1\n" \
  "buffer_load_dwordx4 v[144:147], %[ao], %[rs], s55 offen sc0 sc1\n" \
  "buffer_load_dwordx4 v[148:151], %[ao], %[rs], s55 offen offset:1024 sc0 sc1\n" \
  "s_waitcnt vmcnt(6)\n" \
  "v_min3_u32 v126, v184, v185, v186\n" \
  "v_min3_u32 v125, v187, v188, v189\n" \
  "v_min3_u32 v126, v126, v190, v191\n" \
  "v_min_u32 v126, v126, v125\n" \
  "v_cmp_eq_u32 vcc, 0, v126\n" \
  "s_cbranch_vccz Lg6_%=\n" \
  "s_sleep 2\n" \
  "s_branch Lsll7_%=\n" \
  "Ltk_%=:\n" \
  "s_memrealtime s[74:75]\n" \
  "s_waitcnt lgkmcnt(0)\n" \
  "v_mov_b32 v14, s74\n" \
  "v_mov_b32 v15, s75\n" \
  "v_cmp_lt_u64 vcc, %[td], v[14:15]\n" \
  "s_cbranch_vccnz Lto_%=\n" \
  "s_setpc_b64 s[70:71]\n" \
  "Lto_%=:\n" \
  "s_mov_b32 %4, 1\n" \
  "s_branch Ldone_%=\n" \
  "Lwait_%=:\n" \
  "ds_read_b96 v[120:122], %[sy]\n" \
  "s_add_u32 s63, s63, 1\n" \
  "s_and_b32 s64, s63, 0xff\n" \
  "s_cbranch_scc0 Ltick_%=\n" \
  "Lwait2_%=:\n" \
  "s_waitcnt lgkmcnt(0)\n" \
  "v_min3_u32 v120, v120, v121, v122\n" \
  "s_nop 0\n" \
  "v_readfirstlane_b32 s61, v120\n" \
  "s_cmp_ge_i32 s60, s61\n" \
  "s_cbranch_scc1 Lwait_%=\n" \
  "s_setpc_b64 s[70:71]\n" \
  "Ltick_%=:\n" \
  "s_memrealtime s[74:75]\n" \
  "s_waitcnt lgkmcnt(0)\n" \
  "v_mov_b32 v14, s74\n" \
  "v_mov_b32 v15, s75\n" \
  "v_cmp_lt_u64 vcc, %[td], v[14:15]\n" \
  "s_cbranch_vccz Lwait2_%=\n" \
  "s_mov_b32 %4, 1\n" \
  "s_branch Ldone_%=\n" \
  "Lfin_%=:\n" \
  "v_and_b32 v112, 0xffff7fff, v112\n" \
  "v_and_b32 v113, 0xffff7fff, v113\n" \
  "v_and_b32 v114, 0xffff7fff, v114\n" \
  "v_and_b32 v115, 0xffff7fff, v115\n" \
  "v_and_b32 v116, 0xffff7fff, v116\n" \
  "v_and_b32 v117, 0xffff7fff, v117\n" \
  "v_and_b32 v118, 0xffff7fff, v118\n" \
  "v_and_b32 v119, 0xffff7fff, v119\n" \
  "s_mov_b32 s60, 21\n" \
  "s_cmp_lt_i32 s60, s61\n" \
  "s_cbranch_scc1 Lpa16_%=\n" \
  "s_swappc_b64 s[70:71], s[72:73]\n" \
  "Lpa16_%=:\n" \
  "s_waitcnt lgkmcnt(9)\n" \
  "v_mfma_f32_32x32x64_f8f6f4 v[48:63], v[160:167], v[192:199], v[48:63]\n" \
  "ds_read_b96 v[120:122], %[sy]\n" \
  "v_readlane_b32 s56, %[tab], 20\n" \
  "v_readlane_b32 s57, %[tab], 21\n" \
  "ds_read_b64_tr_b8 v[224:225], v116 offset:16384\n" \
  "ds_read_b64_tr_b8 v[226:227], v116 offset:17408\n" \
  "ds_read_b64_tr_b8 v[228:229], v116 offset:18432\n" \
  "ds_read_b64_tr_b8 v[230:231], v116 offset:19456\n" \
  "v_mfma_f32_32x32x64_f8f6f4 v[64:79], v[160:167], v[200:207], v[64:79]\n" \
  "ds_read_b64_tr_b8 v[232:233], v117 offset:16384\n" \
  "ds_read_b64_tr_b8 v[234:235], v117 offset:17408\n" \
  "ds_read_b64_tr_b8 v[236:237], v117 offset:18432\n" \
  "ds_read_b64_tr_b8 v[238:239], v117 offset:19456\n" \
  "buffer_load_dwordx4 v[128:131], %[ao], %[rs], s56 offen sc0 sc1\n" \
  "buffer_load_dwordx4 v[132:135], %[ao], %[rs], s56 offen offset:1024 sc0 sc1\n" \
  "s_waitcnt lgkmcnt(8)\n" \
  "v_mfma_f32_32x32x64_f8f6f4 v[80:95], v[160:167], v[208:215], v[80:95]\n" \
  "v_min3_u32 v120, v120, v121, v122\n" \
  "ds_read_b64_tr_b8 v[240:241], v118 offset:16384\n" \
  "ds_read_b64_tr_b8 v[242:243], v118 offset:17408\n" \
  "ds_read_b64_tr_b8 v[244:245], v118 offset:18432\n" \
  "ds_read_b64_tr_b8 v[246:247], v118 offset:19456\n" \
  "buffer_load_dwordx4 v[136:139], %[ao], %[rs], s57 offen sc0 sc1\n" \
  "buffer_load_dwordx4 v[140:143], %[ao], %[rs], s57 offen offset:1024 sc0 sc1\n" \
  "v_readfirstlane_b32 s61, v120\n" \
  "v_mfma_f32_32x32x64_f8f6f4 v[96:111], v[160:167], v[216:223], v[96:111]\n" \
  "ds_read_b64_tr_b8 v[248:249], v119 offset:16384\n" \
  "ds_read_b64_tr_b8 v[250:251], v119 offset:17408\n" \
  "ds_read_b64_tr_b8 v[252:253], v119 offset:18432\n" \
  "ds_read_b64_tr_b8 v[254:255], v119 offset:19456\n" \
  "s_waitcnt vmcnt(8)\n" \
  "v_min3_u32 v126, v168, v169, v170\n" \
  "v_min3_u32 v125, v171, v172, v173\n" \
  "v_min3_u32 v126, v126, v174, v175\n" \
  "v_min_u32 v126, v126, v125\n" \
  "v_cmp_eq_u32 vcc, 0, v126\n" \
  "s_cbranch_vccnz Lfs17_%=\n" \
  "Lfr17_%=:\n" \
  "s_mov_b32 s60, 23\n" \
  "s_cmp_lt_i32 s60, s61\n" \
  "s_cbranch_scc1 Lpa17_%=\n" \
  "s_swappc_b64 s[70:71], s[72:73]\n" \
  "Lpa17_%=:\n" \
  "s_waitcnt lgkmcnt(8)\n" \
  "v_mfma_f32_32x32x64_f8f6f4 v[48:63], v[168:175], v[224:231], v[48:63]\n" \
  "ds_read_b96 v[120:122], %[sy]\n" \
  "v_readlane_b32 s56, %[tab], 22\n" \
  "v_readlane_b32 s57, %[tab], 23\n" \
  "ds_read_b64_tr_b8 v[192:193], v112 offset:24576\n" \
  "ds_read_b64_tr_b8 v[194:195], v112 offset:25600\n" \
  "ds_read_b64_tr_b8 v[196:197], v112 offset:26624\n" \
  "ds_read_b64_tr_b8 v[198:199], v112 offset:27648\n" \
  "v_mfma_f32_32x32x64_f8f6f4 v[64:79], v[168:175], v[232:239], v[64:79]\n" \
  "ds_read_b64_tr_b8 v[200:201], v113 offset:24576\n" \
  "ds_read_b64_tr_b8 v[202:203], v113 offset:25600\n" \
  "ds_read_b64_tr_b8 v[204:205], v113 offset:26624\n" \
  "ds_read_b64_tr_b8 v[206:207], v113 offset:27648\n" \
  "buffer_load_dwordx4 v[144:147], %[ao], %[rs], s56 offen sc0 sc1\n" \
  "buffer_load_dwordx4 v[148:151], %[ao], %[rs], s56 offen offset:1024 sc0 sc1\n" \
  "s_waitcnt lgkmcnt(8)\n" \
  "v_mfma_f32_32x32x64_f8f6f4 v[80:95], v[168:175], v[240:247], v[80:95]\n" \
  "v_min3_u32 v120, v120, v121, v122\n" \
  "ds_read_b64_tr_b8 v[208:209], v114 offset:24576\n" \
  "ds_read_b64_tr_b8 v[210:211], v114 offset:25600\n" \
  "ds_read_b64_tr_b8 v[212:213], v114 offset:26624\n" \
  "ds_read_b64_tr_b8 v[214:215], v114 offset:27648\n" \
  "buffer_load_dwordx4 v[152:155], %[ao], %[rs], s57 offen sc0 sc1\n" \
  "buffer_load_dwordx4 v[156:159], %[ao], %[rs], s57 offen offset:1024 sc0 sc1\n" \
  "v_readfirstlane_b32 s61, v120\n" \
  "v_mfma_f32_32x32x64_f8f6f4 v[96:111], v[168:175], v[248:255], v[96:111]\n" \
  "ds_read_b64_tr_b8 v[216:217], v115 offset:24576\n" \
  "ds_read_b64_tr_b8 v[218:219], v115 offset:25600\n" \
  "ds_read_b64_tr_b8 v[220:221], v115 offset:26624\n" \
  "ds_read_b64_tr_b8 v[222:223], v115 offset:27648\n" \
  "s_waitcnt vmcnt(10)\n" \
  "v_min3_u32 v126, v176, v177, v178\n" \
  "v_min3_u32 v125, v179, v180, v181\n" \
  "v_min3_u32 v126, v126, v182, v183\n" \
  "v_min_u32 v126, v126, v125\n" \
  "v_cmp_eq_u32 vcc, 0, v126\n" \
  "s_cbranch_vccnz Lfs18_%=\n" \
  "Lfr18_%=:\n" \
  "s_mov_b32 s60, 25\n" \
  "s_cmp_lt_i32 s60, s61\n" \
  "s_cbranch_scc1 Lpa18_%=\n" \
  "s_swappc_b64 s[70:71], s[72:73]\n" \
  "Lpa18_%=:\n" \
  "s_waitcnt lgkmcnt(8)\n" \
  "v_mfma_f32_32x32x64_f8f6f4 v[48:63], v[176:183], v[192:199], v[48:63]\n" \
  "ds_read_b96 v[120:122], %[sy]\n" \
  "v_readlane_b32 s56, %[tab], 24\n" \
  "v_readlane_b32 s57, %[tab], 25\n" \
  "ds_read_b64_tr_b8 v[224:225], v116 offset:24576\n" \
  "ds_read_b64_tr_b8 v[226:227], v116 offset:25600\n" \
  "ds_read_b64_tr_b8 v[228:229], v116 offset:26624\n" \
  "ds_read_b64_tr_b8 v[230:231], v116 offset:27648\n" \
  "v_mfma_f32_32x32x64_f8f6f4 v[64:79], v[176:183], v[200:207], v[64:79]\n" \
  "ds_read_b64_tr_b8 v[232:233], v117 offset:24576\n" \
  "ds_read_b64_tr_b8 v[234:235], v117 offset:25600\n" \
  "ds_read_b64_tr_b8 v[236:237], v117 offset:26624\n" \
  "ds_read_b64_tr_b8 v[238:239], v117 offset:27648\n" \
  "buffer_load_dwordx4 v[16:19], %[ao], %[rs], s56 offen sc0 sc1\n" \
  "buffer_load_dwordx4 v[20:23], %[ao], %[rs], s56 offen offset:1024 sc0 sc1\n" \
  "s_waitcnt lgkmcnt(8)\n" \
  "v_mfma_f32_32x32x64_f8f6f4 v[80:95], v[176:183], v[208:215], v[80:95]\n" \
  "v_min3_u32 v120, v120, v121, v122\n" \
  "ds_read_b64_tr_b8 v[240:241], v118 offset:24576\n" \
  "ds_read_b64_tr_b8 v[242:243], v118 offset:25600\n" \
  "ds_read_b64_tr_b8 v[244:245], v118 offset:26624\n" \
  "ds_read_b64_tr_b8 v[246:247], v118 offset:27648\n" \
  "buffer_load_dwordx4 v[24:27], %[ao], %[rs], s57 offen sc0 sc1\n" \
  "buffer_load_dwordx4 v[28:31], %[ao], %[rs], s57 offen offset:1024 sc0 sc1\n" \
  "v_readfirstlane_b32 s61, v120\n" \
  "v_mfma_f32_32x32x64_f8f6f4 v[96:111], v[176:183], v[216:223], v[96:111]\n" \
  "ds_read_b64_tr_b8 v[248:249], v119 offset:24576\n" \
  "ds_read_b64_tr_b8 v[250:251], v119 offset:25600\n" \
  "ds_read_b64_tr_b8 v[252:253], v119 offset:26624\n" \
  "ds_read_b64_tr_b8 v[254:255], v119 offset:27648\n" \
  "s_waitcnt vmcnt(12)\n" \
  "v_min3_u32 v126, v184, v185, v186\n" \
  "v_min3_u32 v125, v187, v188, v189\n" \
  "v_min3_u32 v126, v126, v190, v191\n" \
  "v_min_u32 v126, v126, v125\n" \
  "v_cmp_eq_u32 vcc, 0, v126\n" \
  "s_cbranch_vccnz Lfs19_%=\n" \
  "Lfr19_%=:\n" \
  "s_mov_b32 s60, 27\n" \
  "s_cmp_lt_i32 s60, s61\n" \
  "s_cbranch_scc1 Lpa19_%=\n" \
  "s_swappc_b64 s[70:71], s[72:73]\n" \
  "Lpa19_%=:\n" \
  "s_waitcnt lgkmcnt(8)\n" \
  "v_mfma_f32_32x32x64_f8f6f4 v[48:63], v[184:191], v[224:231], v[48:63]\n" \
  "ds_read_b96 v[120:122], %[sy]\n" \
  "v_readlane_b32 s56, %[tab], 26\n" \
  "v_readlane_b32 s57, %[tab], 27\n" \
  "ds_read_b64_tr_b8 v[208:209], v114 offset:32768\n" \
  "ds_read_b64_tr_b8 v[210:211], v114 offset:33792\n" \
  "ds_read_b64_tr_b8 v[212:213], v114 offset:34816\n" \
  "ds_read_b64_tr_b8 v[214:215], v114 offset:35840\n" \
  "v_mfma_f32_32x32x64_f8f6f4 v[64:79], v[184:191], v[232:239], v[64:79]\n" \
  "ds_read_b64_tr_b8 v[216:217], v115 offset:32768\n" \
  "ds_read_b64_tr_b8 v[218:219], v115 offset:33792\n" \
  "ds_read_b64_tr_b8 v[220:221], v115 offset:34816\n" \
  "ds_read_b64_tr_b8 v[222:223], v115 offset:35840\n" \
  "buffer_load_dwordx4 v[32:35], %[ao], %[rs], s56 offen sc0 sc1\n" \
  "buffer_load_dwordx4 v[36:39], %[ao], %[rs], s56 offen offset:1024 sc0 sc1\n" \
  "s_waitcnt lgkmcnt(8)\n" \
  "v_mfma_f32_32x32x64_f8f6f4 v[80:95], v[184:191], v[240:247], v[80:95]\n" \
  "v_min3_u32 v120, v120, v121, v122\n" \
  "ds_read_b64_tr_b8 v[192:193], v118 offset:32768\n" \
  "ds_read_b64_tr_b8 v[194:195], v118 offset:33792\n" \
  "ds_read_b64_tr_b8 v[196:197], v118 offset:34816\n" \
  "ds_read_b64_tr_b8 v[198:199], v118 offset:35840\n" \
  "buffer_load_dwordx4 v[40:43], %[ao], %[rs], s57 offen sc0 sc1\n" \
  "buffer_load_dwordx4 v[44:47], %[ao], %[rs], s57 offen offset:1024 sc0 sc1\n" \
  "v_readfirstlane_b32 s61, v120\n" \
  "v_mfma_f32_32x32x64_f8f6f4 v[96:111], v[184:191], v[248:255], v[96:111]\n" \
  "ds_read_b64_tr_b8 v[200:201], v119 offset:32768\n" \
  "ds_read_b64_tr_b8 v[202:203], v119 offset:33792\n" \
  "ds_read_b64_tr_b8 v[204:205], v119 offset:34816\n" \
  "ds_read_b64_tr_b8 v[206:207], v119 offset:35840\n" \
  "s_waitcnt vmcnt(14)\n" \
  "v_min3_u32 v126, v128, v129, v130\n" \
  "v_min3_u32 v125, v131, v132, v133\n" \
  "v_min3_u32 v126, v126, v134, v135\n" \
  "v_min_u32 v126, v126, v125\n" \
  "v_cmp_eq_u32 vcc, 0, v126\n" \
  "s_cbranch_vccnz Lfs20_%=\n" \
  "Lfr20_%=:\n" \
  "s_mov_b32 s69, 0\n" \
  "s_mov_b32 s60, 30\n" \
  "s_cmp_lt_i32 s60, s61\n" \
  "s_cbranch_scc1 Lfa_%=\n" \
  "s_swappc_b64 s[70:71], s[72:73]\n" \
  "Lfa_%=:\n" \
  "s_waitcnt lgkmcnt(8)\n" \
  "v_mfma_f32_32x32x64_f8f6f4 v[80:95], v[128:135], v[208:215], v[80:95]\n" \
  "ds_read_b64_tr_b8 v[240:241], v114 offset:40960\n" \
  "ds_read_b64_tr_b8 v[242:243], v114 offset:41984\n" \
  "ds_read_b64_tr_b8 v[244:245], v114 offset:43008\n" \
  "ds_read_b64_tr_b8 v[246:247], v114 offset:44032\n" \
  "v_readlane_b32 s56, %[tab], 30\n" \
  "v_readlane_b32 s57, %[tab], 31\n" \
  "v_readlane_b32 s58, %[tab], 28\n" \
  "v_readlane_b32 s59, %[tab], 29\n" \
  "s_nop 4\n" \
  "buffer_load_dwordx4 v[176:179], %[ao], %[rs], s56 offen sc0 sc1\n" \
  "buffer_load_dwordx4 v[180:183], %[ao], %[rs], s56 offen offset:1024 sc0 sc1\n" \
  "buffer_load_dwordx4 v[184:187], %[ao], %[rs], s57 offen sc0 sc1\n" \
  "buffer_load_dwordx4 v[188:191], %[ao], %[rs], s57 offen offset:1024 sc0 sc1\n" \
  "buffer_load_dwordx4 v[160:163], %[ao], %[rs], s58 offen sc0 sc1\n" \
  "buffer_load_dwordx4 v[164:167], %[ao], %[rs], s58 offen offset:1024 sc0 sc1\n" \
  "buffer_load_dwordx4 v[168:171], %[ao], %[rs], s59 offen sc0 sc1\n" \
  "buffer_load_dwordx4 v[172:175], %[ao], %[rs], s59 offen offset:1024 sc0 sc1\n" \
  "v_mfma_f32_32x32x64_f8f6f4 v[96:111], v[128:135], v[216:223], v[96:111]\n" \
  "ds_read_b64_tr_b8 v[248:249], v115 offset:40960\n" \
  "ds_read_b64_tr_b8 v[250:251], v115 offset:41984\n" \
  "ds_read_b64_tr_b8 v[252:253], v115 offset:43008\n" \
  "ds_read_b64_tr_b8 v[254:255], v115 offset:44032\n" \
  "s_waitcnt vmcnt(20)\n" \
  "v_min3_u32 v126, v136, v137, v138\n" \
  "v_min3_u32 v125, v139, v140, v141\n" \
  "v_min3_u32 v126, v126, v142, v143\n" \
  "v_min_u32 v126, v126, v125\n" \
  "v_cmp_eq_u32 vcc, 0, v126\n" \
  "s_cbranch_vccnz Lfs21_%=\n" \
  "Lfr21_%=:\n" \
  "s_waitcnt lgkmcnt(8)\n" \
  "v_mfma_f32_32x32x64_f8f6f4 v[80:95], v[136:143], v[192:199], v[80:95]\n" \
  "ds_read_b64_tr_b8 v[224:225], v118 offset:40960\n" \
  "ds_read_b64_tr_b8 v[226:227], v118 offset:41984\n" \
  "ds_read_b64_tr_b8 v[228:229], v118 offset:43008\n" \
  "ds_read_b64_tr_b8 v[230:231], v118 offset:44032\n" \
  "v_mfma_f32_32x32x64_f8f6f4 v[96:111], v[136:143], v[200:207], v[96:111]\n" \
  "ds_read_b64_tr_b8 v[232:233], v119 offset:40960\n" \
  "ds_read_b64_tr_b8 v[234:235], v119 offset:41984\n" \
  "ds_read_b64_tr_b8 v[236:237], v119 offset:43008\n" \
  "ds_read_b64_tr_b8 v[238:239], v119 offset:44032\n" \
  "s_waitcnt vmcnt(18)\n" \
  "v_min3_u32 v126, v144, v145, v146\n" \
  "v_min3_u32 v125, v147, v148, v149\n" \
  "v_min3_u32 v126, v126, v150, v151\n" \
  "v_min_u32 v126, v126, v125\n" \
  "v_cmp_eq_u32 vcc, 0, v126\n" \
  "s_cbranch_vccnz Lfs22_%=\n" \
  "Lfr22_%=:\n" \
  "s_waitcnt lgkmcnt(8)\n" \
  "v_mfma_f32_32x32x64_f8f6f4 v[80:95], v[144:151], v[240:247], v[80:95]\n" \
  "ds_read_b64_tr_b8 v[208:209], v114 offset:49152\n" \
  "ds_read_b64_tr_b8 v[210:211], v114 offset:50176\n" \
  "ds_read_b64_tr_b8 v[212:213], v114 offset:51200\n" \
  "ds_read_b64_tr_b8 v[214:215], v114 offset:52224\n" \
  "v_mfma_f32_32x32x64_f8f6f4 v[96:111], v[144:151], v[248:255], v[96:111]\n" \
  "ds_read_b64_tr_b8 v[216:217], v115 offset:49152\n" \
  "ds_read_b64_tr_b8 v[218:219], v115 offset:50176\n" \
  "ds_read_b64_tr_b8 v[220:221], v115 offset:51200\n" \
  "ds_read_b64_tr_b8 v[222:223], v115 offset:52224\n" \
  "s_waitcnt vmcnt(16)\n" \
  "v_min3_u32 v126, v152, v153, v154\n" \
  "v_min3_u32 v125, v155, v156, v157\n" \
  "v_min3_u32 v126, v126, v158, v159\n" \
  "v_min_u32 v126, v126, v125\n" \
  "v_cmp_eq_u32 vcc, 0, v126\n" \
  "s_cbranch_vccnz Lfs23_%=\n" \
  "Lfr23_%=:\n" \
  "s_waitcnt lgkmcnt(8)\n" \
  "v_mfma_f32_32x32x64_f8f6f4 v[80:95], v[152:159], v[224:231], v[80:95]\n" \
  "ds_read_b64_tr_b8 v[192:193], v118 offset:49152\n" \
  "ds_read_b64_tr_b8 v[194:195], v118 offset:50176\n" \
  "ds_read_b64_tr_b8 v[196:197], v118 offset:51200\n" \
  "ds_read_b64_tr_b8 v[198:199], v118 offset:52224\n" \
  "v_mfma_f32_32x32x64_f8f6f4 v[96:111], v[152:159], v[232:239], v[96:111]\n" \
  "ds_read_b64_tr_b8 v[200:201], v119 offset:49152\n" \
  "ds_read_b64_tr_b8 v[202:203], v119 offset:50176\n" \
  "ds_read_b64_tr_b8 v[204:205], v119 offset:51200\n" \
  "ds_read_b64_tr_b8 v[206:207], v119 offset:52224\n" \
  "s_waitcnt vmcnt(14)\n" \
  "v_min3_u32 v126, v16, v17, v18\n" \
  "v_min3_u32 v125, v19, v20, v21\n" \
  "v_min3_u32 v126, v126, v22, v23\n" \
  "v_min_u32 v126, v126, v125\n" \
  "v_cmp_eq_u32 vcc, 0, v126\n" \
  "s_cbranch_vccnz Lfs24_%=\n" \
  "Lfr24_%=:\n" \
  "s_waitcnt lgkmcnt(8)\n" \
  "v_mfma_f32_32x32x64_f8f6f4 v[80:95], v[16:23], v[208:215], v[80:95]\n" \
  "ds_read_b64_tr_b8 v[240:241], v114 offset:57344\n" \
  "ds_read_b64_tr_b8 v[242:243], v114 offset:58368\n" \
  "ds_read_b64_tr_b8 v[244:245], v114 offset:59392\n" \
  "ds_read_b64_tr_b8 v[246:247], v114 offset:60416\n" \
  "v_mfma_f32_32x32x64_f8f6f4 v[96:111], v[16:23], v[216:223], v[96:111]\n" \
  "ds_read_b64_tr_b8 v[248:249], v115 offset:57344\n" \
  "ds_read_b64_tr_b8 v[250:251], v115 offset:58368\n" \
  "ds_read_b64_tr_b8 v[252:253], v115 offset:59392\n" \
  "ds_read_b64_tr_b8 v[254:255], v115 offset:60416\n" \
  "s_waitcnt vmcnt(12)\n" \
  "v_min3_u32 v126, v24, v25, v26\n" \
  "v_min3_u32 v125, v27, v28, v29\n" \
  "v_min3_u32 v126, v126, v30, v31\n" \
  "v_min_u32 v126, v126, v125\n" \
  "v_cmp_eq_u32 vcc, 0, v126\n" \
  "s_cbranch_vccnz Lfs25_%=\n" \
  "Lfr25_%=:\n" \
  "s_waitcnt lgkmcnt(8)\n" \
  "v_mfma_f32_32x32x64_f8f6f4 v[80:95], v[24:31], v[192:199], v[80:95]\n" \
  "ds_read_b64_tr_b8 v[224:225], v118 offset:57344\n" \
  "ds_read_b64_tr_b8 v[226:227], v118 offset:58368\n" \
  "ds_read_b64_tr_b8 v[228:229], v118 offset:59392\n" \
  "ds_read_b64_tr_b8 v[230:231], v118 offset:60416\n" \
  "v_mfma_f32_32x32x64_f8f6f4 v[96:111], v[24:31], v[200:207], v[96:111]\n" \
  "ds_read_b64_tr_b8 v[232:233], v119 offset:57344\n" \
  "ds_read_b64_tr_b8 v[234:235], v119 offset:58368\n" \
  "ds_read_b64_tr_b8 v[236:237], v119 offset:59392\n" \
  "ds_read_b64_tr_b8 v[238:239], v119 offset:60416\n" \
  "s_waitcnt vmcnt(10)\n" \
  "v_min3_u32 v126, v32, v33, v34\n" \
  "v_min3_u32 v125, v35, v36, v37\n" \
  "v_min3_u32 v126, v126, v38, v39\n" \
  "v_min_u32 v126, v126, v125\n" \
  "v_cmp_eq_u32 vcc, 0, v126\n" \
  "s_cbranch_vccnz Lfs26_%=\n" \
  "Lfr26_%=:\n" \
  "s_waitcnt lgkmcnt(8)\n" \
  "v_mfma_f32_32x32x64_f8f6f4 v[80:95], v[32:39], v[240:247], v[80:95]\n" \
  "ds_read_b96 v[120:122], %[sy]\n" \
  "v_add_u32 v12, 140288, v114\n" \
  "ds_read_b64_tr_b8 v[208:209], v12 offset:0\n" \
  "ds_read_b64_tr_b8 v[210:211], v12 offset:1024\n" \
  "ds_read_b64_tr_b8 v[212:213], v12 offset:2048\n" \
  "ds_read_b64_tr_b8 v[214:215], v12 offset:3072\n" \
  "v_mfma_f32_32x32x64_f8f6f4 v[96:111], v[32:39], v[248:255], v[96:111]\n" \
  "v_add_u32 v13, 140288, v115\n" \
  "ds_read_b64_tr_b8 v[216:217], v13 offset:0\n" \
  "ds_read_b64_tr_b8 v[218:219], v13 offset:1024\n" \
  "ds_read_b64_tr_b8 v[220:221], v13 offset:2048\n" \
  "ds_read_b64_tr_b8 v[222:223], v13 offset:3072\n" \
  "s_waitcnt vmcnt(8)\n" \
  "v_min3_u32 v126, v40, v41, v42\n" \
  "v_min3_u32 v125, v43, v44, v45\n" \
  "v_min3_u32 v126, v126, v46, v47\n" \
  "v_min_u32 v126, v126, v125\n" \
  "v_cmp_eq_u32 vcc, 0, v126\n" \
  "s_cbranch_vccnz Lfs27_%=\n" \
  "Lfr27_%=:\n" \
  "s_waitcnt lgkmcnt(8)\n" \
  "v_mfma_f32_32x32x64_f8f6f4 v[80:95], v[40:47], v[224:231], v[80:95]\n" \
  "v_min3_u32 v120, v120, v121, v122\n" \
  "ds_read_b64_tr_b8 v[192:193], v12 offset:8192\n" \
  "ds_read_b64_tr_b8 v[194:195], v12 offset:9216\n" \
  "ds_read_b64_tr_b8 v[196:197], v12 offset:10240\n" \
  "ds_read_b64_tr_b8 v[198:199], v12 offset:11264\n" \
  "v_readfirstlane_b32 s61, v120\n" \
  "v_mfma_f32_32x32x64_f8f6f4 v[96:111], v[40:47], v[232:239], v[96:111]\n" \
  "ds_read_b64_tr_b8 v[200:201], v13 offset:8192\n" \
  "ds_read_b64_tr_b8 v[202:203], v13 offset:9216\n" \
  "ds_read_b64_tr_b8 v[204:205], v13 offset:10240\n" \
  "ds_read_b64_tr_b8 v[206:207], v13 offset:11264\n" \
  "s_waitcnt vmcnt(6)\n" \
  "v_min3_u32 v126, v176, v177, v178\n" \
  "v_min3_u32 v125, v179, v180, v181\n" \
  "v_min3_u32 v126, v126, v182, v183\n" \
  "v_min_u32 v126, v126, v125\n" \
  "v_cmp_eq_u32 vcc, 0, v126\n" \
  "s_cbranch_vccnz Lfs30_%=\n" \
  "Lfr30_%=:\n" \
  "s_mov_b32 s60, 50\n" \
  "s_cmp_lt_i32 s60, s61\n" \
  "s_cbranch_scc1 Lfd_%=\n" \
  "s_swappc_b64 s[70:71], s[72:73]\n" \
  "Lfd_%=:\n" \
  "s_waitcnt lgkmcnt(8)\n" \
  "v_mfma_f32_32x32x64_f8f6f4 v[80:95], v[176:183], v[208:215], v[80:95]\n" \
  "ds_read_b64_tr_b8 v[240:241], v114 offset:0\n" \
  "ds_read_b64_tr_b8 v[242:243], v114 offset:1024\n" \
  "ds_read_b64_tr_b8 v[244:245], v114 offset:2048\n" \
  "ds_read_b64_tr_b8 v[246:247], v114 offset:3072\n" \
  "v_mfma_f32_32x32x64_f8f6f4 v[96:111], v[176:183], v[216:223], v[96:111]\n" \
  "ds_read_b64_tr_b8 v[248:249], v115 offset:0\n" \
  "ds_read_b64_tr_b8 v[250:251], v115 offset:1024\n" \
  "ds_read_b64_tr_b8 v[252:253], v115 offset:2048\n" \
  "ds_read_b64_tr_b8 v[254:255], v115 offset:3072\n" \
  "s_waitcnt vmcnt(4)\n" \
  "v_min3_u32 v126, v184, v185, v186\n" \
  "v_min3_u32 v125, v187, v188, v189\n" \
  "v_min3_u32 v126, v126, v190, v191\n" \
  "v_min_u32 v126, v126, v125\n" \
  "v_cmp_eq_u32 vcc, 0, v126\n" \
  "s_cbranch_vccnz Lfs31_%=\n" \
  "Lfr31_%=:\n" \
  "s_waitcnt lgkmcnt(8)\n" \
  "v_mfma_f32_32x32x64_f8f6f4 v[80:95], v[184:191], v[192:199], v[80:95]\n" \
  "ds_read_b64_tr_b8 v[224:225], v118 offset:0\n" \
  "ds_read_b64_tr_b8 v[226:227], v118 offset:1024\n" \
  "ds_read_b64_tr_b8 v[228:229], v118 offset:2048\n" \
  "ds_read_b64_tr_b8 v[230:231], v118 offset:3072\n" \
  "v_mfma_f32_32x32x64_f8f6f4 v[96:111], v[184:191], v[200:207], v[96:111]\n" \
  "ds_read_b64_tr_b8 v[232:233], v119 offset:0\n" \
  "ds_read_b64_tr_b8 v[234:235], v119 offset:1024\n" \
  "ds_read_b64_tr_b8 v[236:237], v119 offset:2048\n" \
  "ds_read_b64_tr_b8 v[238:239], v119 offset:3072\n" \
  "s_waitcnt vmcnt(2)\n" \
  "v_min3_u32 v126, v160, v161, v162\n" \
  "v_min3_u32 v125, v163, v164, v165\n" \
  "v_min3_u32 v126, v126, v166, v167\n" \
  "v_min_u32 v126, v126, v125\n" \
  "v_cmp_eq_u32 vcc, 0, v126\n" \
  "s_cbranch_vccnz Lfs28_%=\n" \
  "Lfr28_%=:\n" \
  "s_waitcnt lgkmcnt(8)\n" \
  "v_mfma_f32_32x32x64_f8f6f4 v[80:95], v[160:167], v[240:247], v[80:95]\n" \
  "ds_read_b64_tr_b8 v[208:209], v112 offset:32768\n" \
  "ds_read_b64_tr_b8 v[210:211], v112 offset:33792\n" \
  "ds_read_b64_tr_b8 v[212:213], v112 offset:34816\n" \
  "ds_read_b64_tr_b8 v[214:215], v112 offset:35840\n" \
  "v_mfma_f32_32x32x64_f8f6f4 v[96:111], v[160:167], v[248:255], v[96:111]\n" \
  "ds_read_b64_tr_b8 v[216:217], v113 offset:32768\n" \
  "ds_read_b64_tr_b8 v[218:219], v113 offset:33792\n" \
  "ds_read_b64_tr_b8 v[220:221], v113 offset:34816\n" \
  "ds_read_b64_tr_b8 v[222:223], v113 offset:35840\n" \
  "s_waitcnt vmcnt(0)\n" \
  "v_min3_u32 v126, v168, v169, v170\n" \
  "v_min3_u32 v125, v171, v172, v173\n" \
  "v_min3_u32 v126, v126, v174, v175\n" \
  "v_min_u32 v126, v126, v125\n" \
  "v_cmp_eq_u32 vcc, 0, v126\n" \
  "s_cbranch_vccnz Lfs29_%=\n" \
  "Lfr29_%=:\n" \
  "s_movk_i32 s56, 0x1000\n" \
  "s_waitcnt lgkmcnt(8)\n" \
  "v_mfma_f32_32x32x64_f8f6f4 v[80:95], v[168:175], v[224:231], v[80:95]\n" \
  "ds_read_b64_tr_b8 v[192:193], v116 offset:32768\n" \
  "ds_read_b64_tr_b8 v[194:195], v116 offset:33792\n" \
  "ds_read_b64_tr_b8 v[196:197], v116 offset:34816\n" \
  "ds_read_b64_tr_b8 v[198:199], v116 offset:35840\n" \
  "v_mfma_f32_32x32x64_f8f6f4 v[96:111], v[168:175], v[232:239], v[96:111]\n" \
  "ds_read_b64_tr_b8 v[200:201], v117 offset:32768\n" \
  "ds_read_b64_tr_b8 v[202:203], v117 offset:33792\n" \
  "ds_read_b64_tr_b8 v[204:205], v117 offset:34816\n" \
  "ds_read_b64_tr_b8 v[206:207], v117 offset:35840\n" \
  "s_waitcnt lgkmcnt(8)\n" \
  "v_mfma_f32_32x32x64_f8f6f4 v[48:63], v[128:135], v[208:215], v[48:63]\n" \
  "ds_read_b64_tr_b8 v[240:241], v112 offset:40960\n" \
  "ds_read_b64_tr_b8 v[242:243], v112 offset:41984\n" \
  "ds_read_b64_tr_b8 v[244:245], v112 offset:43008\n" \
  "ds_read_b64_tr_b8 v[246:247], v112 offset:44032\n" \
  "buffer_store_dwordx4 v[80:83], %[ao], %[rp], 0 offen offset:0 sc1\n" \
  "buffer_store_dwordx4 v[84:87], %[ao], %[rp], 0 offen offset:1024 sc1\n" \
  "buffer_store_dwordx4 v[88:91], %[ao], %[rp], 0 offen offset:2048 sc1\n" \
  "buffer_store_dwordx4 v[92:95], %[ao], %[rp], 0 offen offset:3072 sc1\n" \
  "v_mfma_f32_32x32x64_f8f6f4 v[64:79], v[128:135], v[216:223], v[64:79]\n" \
  "ds_read_b64_tr_b8 v[248:249], v113 offset:40960\n" \
  "ds_read_b64_tr_b8 v[250:251], v113 offset:41984\n" \
  "ds_read_b64_tr_b8 v[252:253], v113 offset:43008\n" \
  "ds_read_b64_tr_b8 v[254:255], v113 offset:44032\n" \
  "buffer_store_dwordx4 v[96:99], %[ao], %[rp], s56 offen offset:0 sc1\n" \
  "buffer_store_dwordx4 v[100:103], %[ao], %[rp], s56 offen offset:1024 sc1\n" \
  "buffer_store_dwordx4 v[104:107], %[ao], %[rp], s56 offen offset:2048 sc1\n" \
  "buffer_store_dwordx4 v[108:111], %[ao], %[rp], s56 offen offset:3072 sc1\n" \
  "v_mov_b32 v126, %[cl]\n" \
  "s_waitcnt lgkmcnt(8)\n" \
  "v_mfma_f32_32x32x64_f8f6f4 v[48:63], v[136:143], v[192:199], v[48:63]\n" \
  "ds_read_b64_tr_b8 v[224:225], v116 offset:40960\n" \
  "ds_read_b64_tr_b8 v[226:227], v116 offset:41984\n" \
  "ds_read_b64_tr_b8 v[228:229], v116 offset:43008\n" \
  "ds_read_b64_tr_b8 v[230:231], v116 offset:44032\n" \
  "v_mfma_f32_32x32x64_f8f6f4 v[64:79], v[136:143], v[200:207], v[64:79]\n" \
  "ds_read_b64_tr_b8 v[232:233], v117 offset:40960\n" \
  "ds_read_b64_tr_b8 v[234:235], v117 offset:41984\n" \
  "ds_read_b64_tr_b8 v[236:237], v117 offset:43008\n" \
  "ds_read_b64_tr_b8 v[238:239], v117 offset:44032\n" \
  "s_waitcnt lgkmcnt(8)\n" \
  "v_mfma_f32_32x32x64_f8f6f4 v[48:63], v[144:151], v[240:247], v[48:63]\n" \
  "ds_read_b64_tr_b8 v[208:209], v112 offset:49152\n" \
  "ds_read_b64_tr_b8 v[210:211], v112 offset:50176\n" \
  "ds_read_b64_tr_b8 v[212:213], v112 offset:51200\n" \
  "ds_read_b64_tr_b8 v[214:215], v112 offset:52224\n" \
  "v_mfma_f32_32x32x64_f8f6f4 v[64:79], v[144:151], v[248:255], v[64:79]\n" \
  "ds_read_b64_tr_b8 v[216:217], v113 offset:49152\n" \
  "ds_read_b64_tr_b8 v[218:219], v113 offset:50176\n" \
  "ds_read_b64_tr_b8 v[220:221], v113 offset:51200\n" \
  "ds_read_b64_tr_b8 v[222:223], v113 offset:52224\n" \
  "s_waitcnt lgkmcnt(8)\n" \
  "v_mfma_f32_32x32x64_f8f6f4 v[48:63], v[152:159], v[224:231], v[48:63]\n" \
  "ds_read_b64_tr_b8 v[192:193], v116 offset:49152\n" \
  "ds_read_b64_tr_b8 v[194:195], v116 offset:50176\n" \
  "ds_read_b64_tr_b8 v[196:197], v116 offset:51200\n" \
  "ds_read_b64_tr_b8 v[198:199], v116 offset:52224\n" \
  "v_mfma_f32_32x32x64_f8f6f4 v[64:79], v[152:159], v[232:239], v[64:79]\n" \
  "ds_read_b64_tr_b8 v[200:201], v117 offset:49152\n" \
  "ds_read_b64_tr_b8 v[202:203], v117 offset:50176\n" \
  "ds_read_b64_tr_b8 v[204:205], v117 offset:51200\n" \
  "ds_read_b64_tr_b8 v[206:207], v117 offset:52224\n" \
  "s_getreg_b32 s64, hwreg(HW_REG_IB_STS, 0, 4)\n" \
  "s_or_b32 s64, s64, s69\n" \
  "s_cbranch_scc1 Lns7_%=\n" \
  "buffer_store_dword v126, %[ao], %[rf], 0 offen sc1\n" \
  "s_mov_b32 s69, 1\n" \
  "Lns7_%=:\n" \
  "s_waitcnt lgkmcnt(8)\n" \
  "v_mfma_f32_32x32x64_f8f6f4 v[48:63], v[16:23], v[208:215], v[48:63]\n" \
  "ds_read_b64_tr_b8 v[240:241], v112 offset:57344\n" \
  "ds_read_b64_tr_b8 v[242:243], v112 offset:58368\n" \
  "ds_read_b64_tr_b8 v[244:245], v112 offset:59392\n" \
  "ds_read_b64_tr_b8 v[246:247], v112 offset:60416\n" \
  "v_mfma_f32_32x32x64_f8f6f4 v[64:79], v[16:23], v[216:223], v[64:79]\n" \
  "ds_read_b64_tr_b8 v[248:249], v113 offset:57344\n" \
  "ds_read_b64_tr_b8 v[250:251], v113 offset:58368\n" \
  "ds_read_b64_tr_b8 v[252:253], v113 offset:59392\n" \
  "ds_read_b64_tr_b8 v[254:255], v113 offset:60416\n" \
  "s_waitcnt lgkmcnt(8)\n" \
  "v_mfma_f32_32x32x64_f8f6f4 v[48:63], v[24:31], v[192:199], v[48:63]\n" \
  "ds_read_b64_tr_b8 v[224:225], v116 offset:57344\n" \
  "ds_read_b64_tr_b8 v[226:227], v116 offset:58368\n" \
  "ds_read_b64_tr_b8 v[228:229], v116 offset:59392\n" \
  "ds_read_b64_tr_b8 v[230:231], v116 offset:60416\n" \
  "v_mfma_f32_32x32x64_f8f6f4 v[64:79], v[24:31], v[200:207], v[64:79]\n" \
  "ds_read_b64_tr_b8 v[232:233], v117 offset:57344\n" \
  "ds_read_b64_tr_b8 v[234:235], v117 offset:58368\n" \
  "ds_read_b64_tr_b8 v[236:237], v117 offset:59392\n" \
  "ds_read_b64_tr_b8 v[238:239], v117 offset:60416\n" \
  "s_getreg_b32 s64, hwreg(HW_REG_IB_STS, 0, 4)\n" \
  "s_or_b32 s64, s64, s69\n" \
  "s_cbranch_scc1 Lns8_%=\n" \
  "buffer_store_dword v126, %[ao], %[rf], 0 offen sc1\n" \
  "s_mov_b32 s69, 1\n" \
  "Lns8_%=:\n" \
  "s_waitcnt lgkmcnt(8)\n" \
  "v_mfma_f32_32x32x64_f8f6f4 v[48:63], v[32:39], v[240:247], v[48:63]\n" \
  "ds_read_b32 v120, %[dn] offset:32\n" \
  "ds_read_b64_tr_b8 v[208:209], v112 offset:0\n" \
  "ds_read_b64_tr_b8 v[210:211], v112 offset:1024\n" \
  "ds_read_b64_tr_b8 v[212:213], v112 offset:2048\n" \
  "ds_read_b64_tr_b8 v[214:215], v112 offset:3072\n" \
  "v_mfma_f32_32x32x64_f8f6f4 v[64:79], v[32:39], v[248:255], v[64:79]\n" \
  "ds_read_b64_tr_b8 v[216:217], v113 offset:0\n" \
  "ds_read_b64_tr_b8 v[218:219], v113 offset:1024\n" \
  "ds_read_b64_tr_b8 v[220:221], v113 offset:2048\n" \
  "ds_read_b64_tr_b8 v[222:223], v113 offset:3072\n" \
  "s_waitcnt lgkmcnt(8)\n" \
  "v_mfma_f32_32x32x64_f8f6f4 v[48:63], v[40:47], v[224:231], v[48:63]\n" \
  "ds_read_b64_tr_b8 v[192:193], v116 offset:0\n" \
  "ds_read_b64_tr_b8 v[194:195], v116 offset:1024\n" \
  "ds_read_b64_tr_b8 v[196:197], v116 offset:2048\n" \
  "ds_read_b64_tr_b8 v[198:199], v116 offset:3072\n" \
  "v_readfirstlane_b32 s64, v120\n" \
  "s_and_b32 s64, s64, s69\n" \
  "s_cbranch_scc0 Lnf9_%=\n" \
  "buffer_load_dwordx4 v[80:83], %[ao], %[rq], 0 offen offset:0 sc1\n" \
  "buffer_load_dwordx4 v[84:87], %[ao], %[rq], 0 offen offset:1024 sc1\n" \
  "buffer_load_dwordx4 v[88:91], %[ao], %[rq], 0 offen offset:2048 sc1\n" \
  "buffer_load_dwordx4 v[92:95], %[ao], %[rq], 0 offen offset:3072 sc1\n" \
  "s_mov_b32 s69, 2\n" \
  "Lnf9_%=:\n" \
  "v_mfma_f32_32x32x64_f8f6f4 v[64:79], v[40:47], v[232:239], v[64:79]\n" \
  "ds_read_b64_tr_b8 v[200:201], v117 offset:0\n" \
  "ds_read_b64_tr_b8 v[202:203], v117 offset:1024\n" \
  "ds_read_b64_tr_b8 v[204:205], v117 offset:2048\n" \
  "ds_read_b64_tr_b8 v[206:207], v117 offset:3072\n" \
  "s_getreg_b32 s64, hwreg(HW_REG_IB_STS, 0, 4)\n" \
  "s_or_b32 s64, s64, s69\n" \
  "s_cbranch_scc1 Lns9_%=\n" \
  "buffer_store_dword v126, %[ao], %[rf], 0 offen sc1\n" \
  "s_mov_b32 s69, 1\n" \
  "Lns9_%=:\n" \
  "s_waitcnt lgkmcnt(8)\n" \
  "v_mfma_f32_32x32x64_f8f6f4 v[48:63], v[160:167], v[208:215], v[48:63]\n" \
  "ds_read_b32 v120, %[dn] offset:32\n" \
  "v_add_u32 v12, 140288, v112\n" \
  "ds_read_b64_tr_b8 v[240:241], v12 offset:0\n" \
  "ds_read_b64_tr_b8 v[242:243], v12 offset:1024\n" \
  "ds_read_b64_tr_b8 v[244:245], v12 offset:2048\n" \
  "ds_read_b64_tr_b8 v[246:247], v12 offset:3072\n" \
  "v_mfma_f32_32x32x64_f8f6f4 v[64:79], v[160:167], v[216:223], v[64:79]\n" \
  "v_add_u32 v13, 140288, v113\n" \
  "ds_read_b64_tr_b8 v[248:249], v13 offset:0\n" \
  "ds_read_b64_tr_b8 v[250:251], v13 offset:1024\n" \
  "ds_read_b64_tr_b8 v[252:253], v13 offset:2048\n" \
  "ds_read_b64_tr_b8 v[254:255], v13 offset:3072\n" \
  "s_waitcnt lgkmcnt(8)\n" \
  "v_mfma_f32_32x32x64_f8f6f4 v[48:63], v[168:175], v[192:199], v[48:63]\n" \
  "ds_read_b64_tr_b8 v[224:225], v12 offset:8192\n" \
  "ds_read_b64_tr_b8 v[226:227], v12 offset:9216\n" \
  "ds_read_b64_tr_b8 v[228:229], v12 offset:10240\n" \
  "ds_read_b64_tr_b8 v[230:231], v12 offset:11264\n" \
  "v_readfirstlane_b32 s64, v120\n" \
  "s_and_b32 s64, s64, s69\n" \
  "s_cbranch_scc0 Lnf10_%=\n" \
  "buffer_load_dwordx4 v[80:83], %[ao], %[rq], 0 offen offset:0 sc1\n" \
  "buffer_load_dwordx4 v[84:87], %[ao], %[rq], 0 offen offset:1024 sc1\n" \
  "buffer_load_dwordx4 v[88:91], %[ao], %[rq], 0 offen offset:2048 sc1\n" \
  "buffer_load_dwordx4 v[92:95], %[ao], %[rq], 0 offen offset:3072 sc1\n" \
  "s_mov_b32 s69, 2\n" \
  "Lnf10_%=:\n" \
  "v_mfma_f32_32x32x64_f8f6f4 v[64:79], v[168:175], v[200:207], v[64:79]\n" \
  "ds_read_b64_tr_b8 v[232:233], v13 offset:8192\n" \
  "ds_read_b64_tr_b8 v[234:235], v13 offset:9216\n" \
  "ds_read_b64_tr_b8 v[236:237], v13 offset:10240\n" \
  "ds_read_b64_tr_b8 v[238:239], v13 offset:11264\n" \
  "s_getreg_b32 s64, hwreg(HW_REG_IB_STS, 0, 4)\n" \
  "s_or_b32 s64, s64, s69\n" \
  "s_cbranch_scc1 Lns10_%=\n" \
  "buffer_store_dword v126, %[ao], %[rf], 0 offen sc1\n" \
  "s_mov_b32 s69, 1\n" \
  "Lns10_%=:\n" \
  "s_waitcnt lgkmcnt(8)\n" \
  "v_mfma_f32_32x32x64_f8f6f4 v[48:63], v[176:183], v[240:247], v[48:63]\n" \
  "ds_read_b32 v120, %[dn] offset:32\n" \
  "v_mfma_f32_32x32x64_f8f6f4 v[64:79], v[176:183], v[248:255], v[64:79]\n" \
  "s_waitcnt lgkmcnt(0)\n" \
  "v_mfma_f32_32x32x64_f8f6f4 v[48:63], v[184:191], v[224:231], v[48:63]\n" \
  "v_readfirstlane_b32 s64, v120\n" \
  "s_and_b32 s64, s64, s69\n" \
  "s_cbranch_scc0 Lnf11_%=\n" \
  "buffer_load_dwordx4 v[80:83], %[ao], %[rq], 0 offen offset:0 sc1\n" \
  "buffer_load_dwordx4 v[84:87], %[ao], %[rq], 0 offen offset:1024 sc1\n" \
  "buffer_load_dwordx4 v[88:91], %[ao], %[rq], 0 offen offset:2048 sc1\n" \
  "buffer_load_dwordx4 v[92:95], %[ao], %[rq], 0 offen offset:3072 sc1\n" \
  "s_mov_b32 s69, 2\n" \
  "Lnf11_%=:\n" \
  "v_mfma_f32_32x32x64_f8f6f4 v[64:79], v[184:191], v[232:239], v[64:79]\n" \
  "s_cmp_lg_u32 s69, 0\n" \
  "s_cbranch_scc1 Lns11_%=\n" \
  "s_waitcnt vmcnt(0)\n" \
  "buffer_store_dword v126, %[ao], %[rf], 0 offen sc1\n" \
  "s_mov_b32 s69, 1\n" \
  "Lns11_%=:\n" \
  "Lpk_%=:\n" \
  "ds_read_b128 v[120:123], %[sy] offset:32\n" \
  "s_add_u32 s63, s63, 1\n" \
  "s_and_b32 s64, s63, 0xff\n" \
  "s_cbranch_scc1 Lpk2_%=\n" \
  "s_swappc_b64 s[70:71], s[50:51]\n" \
  "Lpk2_%=:\n" \
  "v_add_u32 v12, %[pk], %[ao]\n" \
  "v_mov_b32 v13, 1\n" \
  "s_waitcnt lgkmcnt(0)\n" \
  "v_min3_u32 v120, v120, v121, v122\n" \
  "v_min_u32 v120, v120, v123\n" \
  "s_nop 0\n" \
  "v_readfirstlane_b32 s64, v120\n" \
  "s_cmp_lt_u32 s64, 33\n" \
  "s_cbranch_scc1 Lpk_%=\n" \
  "s_nop 7\n" \
  "ds_write_b128 v12, v[64:67] offset:0\n" \
  "ds_write_b128 v12, v[68:71] offset:1024\n" \
  "ds_write_b128 v12, v[72:75] offset:2048\n" \
  "ds_write_b128 v12, v[76:79] offset:3072\n" \
  "ds_write_b32 %[dn], v13 offset:48\n" \
  "s_cmp_eq_u32 s69, 2\n" \
  "s_cbranch_scc1 Ldone_%=\n" \
  "Lwn_%=:\n" \
  "ds_read_b32 v120, %[dn] offset:32\n" \
  "s_add_u32 s63, s63, 1\n" \
  "s_and_b32 s64, s63, 0xff\n" \
  "s_cbranch_scc1 Lwn2_%=\n" \
  "s_swappc_b64 s[70:71], s[50:51]\n" \
  "Lwn2_%=:\n" \
  "s_waitcnt lgkmcnt(0)\n" \
  "v_readfirstlane_b32 s64, v120\n" \
  "s_cmp_eq_u32 s64, 0\n" \
  "s_cbranch_scc1 Lwn_%=\n" \
  "buffer_load_dwordx4 v[80:83], %[ao], %[rq], 0 offen offset:0 sc1\n" \
  "buffer_load_dwordx4 v[84:87], %[ao], %[rq], 0 offen offset:1024 sc1\n" \
  "buffer_load_dwordx4 v[88:91], %[ao], %[rq], 0 offen offset:2048 sc1\n" \
  "buffer_load_dwordx4 v[92:95], %[ao], %[rq], 0 offen offset:3072 sc1\n" \
  "s_branch Ldone_%=\n" \
  "Lfs17_%=:\n" \
  "s_add_u32 s66, s66, 1\n" \
  "s_and_b32 s64, s66, 0x3f\n" \
  "s_cbranch_scc1 Lfk17_%=\n" \
  "s_swappc_b64 s[70:71], s[50:51]\n" \
  "Lfk17_%=:\n" \
  "v_readlane_b32 s52, %[tab], 17\n" \
  "v_readlane_b32 s53, %[tab], 18\n" \
  "v_readlane_b32 s54, %[tab], 19\n" \
  "v_readlane_b32 s55, %[tab], 20\n" \
  "s_nop 4\n" \
  "buffer_load_dwordx4 v[168:171], %[ao], %[rs], s52 offen sc0 sc1\n" \
  "buffer_load_dwordx4 v[172:175], %[ao], %[rs], s52 offen offset:1024 sc0 sc1\n" \
  "buffer_load_dwordx4 v[176:179], %[ao], %[rs], s53 offen sc0 sc1\n" \
  "buffer_load_dwordx4 v[180:183], %[ao], %[rs], s53 offen offset:1024 sc0 sc1\n" \
  "buffer_load_dwordx4 v[184:187], %[ao], %[rs], s54 offen sc0 sc1\n" \
  "buffer_load_dwordx4 v[188:191], %[ao], %[rs], s54 offen offset:1024 sc0 sc1\n" \
  "buffer_load_dwordx4 v[128:131], %[ao], %[rs], s55 offen sc0 sc1\n" \
  "buffer_load_dwordx4 v[132:135], %[ao], %[rs], s55 offen offset:1024 sc0 sc1\n" \
  "s_waitcnt vmcnt(6)\n" \
  "v_min3_u32 v126, v168, v169, v170\n" \
  "v_min3_u32 v125, v171, v172, v173\n" \
  "v_min3_u32 v126, v126, v174, v175\n" \
  "v_min_u32 v126, v126, v125\n" \
  "v_cmp_eq_u32 vcc, 0, v126\n" \
  "s_cbranch_vccz Lfr17_%=\n" \
  "s_sleep 2\n" \
  "s_branch Lfs17_%=\n" \
  "Lfs18_%=:\n" \
  "s_add_u32 s66, s66, 1\n" \
  "s_and_b32 s64, s66, 0x3f\n" \
  "s_cbranch_scc1 Lfk18_%=\n" \
  "s_swappc_b64 s[70:71], s[50:51]\n" \
  "Lfk18_%=:\n" \
  "v_readlane_b32 s52, %[tab], 18\n" \
  "v_readlane_b32 s53, %[tab], 19\n" \
  "v_readlane_b32 s54, %[tab], 20\n" \
  "v_readlane_b32 s55, %[tab], 21\n" \
  "s_nop 4\n" \
  "buffer_load_dwordx4 v[176:179], %[ao], %[rs], s52 offen sc0 sc1\n" \
  "buffer_load_dwordx4 v[180:183], %[ao], %[rs], s52 offen offset:1024 sc0 sc1\n" \
  "buffer_load_dwordx4 v[184:187], %[ao], %[rs], s53 offen sc0 sc1\n" \
  "buffer_load_dwordx4 v[188:191], %[ao], %[rs], s53 offen offset:1024 sc0 sc1\n" \
  "buffer_load_dwordx4 v[128:131], %[ao], %[rs], s54 offen sc0 sc1\n" \
  "buffer_load_dwordx4 v[132:135], %[ao], %[rs], s54 offen offset:1024 sc0 sc1\n" \
  "buffer_load_dwordx4 v[136:139], %[ao], %[rs], s55 offen sc0 sc1\n" \
  "buffer_load_dwordx4 v[140:143], %[ao], %[rs], s55 offen offset:1024 sc0 sc1\n" \
  "s_waitcnt vmcnt(6)\n" \
  "v_min3_u32 v126, v176, v177, v178\n" \
  "v_min3_u32 v125, v179, v180, v181\n" \
  "v_min3_u32 v126, v126, v182, v183\n" \
  "v_min_u32 v126, v126, v125\n" \
  "v_cmp_eq_u32 vcc, 0, v126\n" \
  "s_cbranch_vccz Lfr18_%=\n" \
  "s_sleep 2\n" \
  "s_branch Lfs18_%=\n" \
  "Lfs19_%=:\n" \
  "s_add_u32 s66, s66, 1\n" \
  "s_and_b32 s64, s66, 0x3f\n" \
  "s_cbranch_scc1 Lfk19_%=\n" \
  "s_swappc_b64 s[70:71], s[50:51]\n" \
  "Lfk19_%=:\n" \
  "v_readlane_b32 s52, %[tab], 19\n" \
  "v_readlane_b32 s53, %[tab], 20\n" \
  "v_readlane_b32 s54, %[tab], 21\n" \
  "v_readlane_b32 s55, %[tab], 22\n" \
  "s_nop 4\n" \
  "buffer_load_dwordx4 v[184:187], %[ao], %[rs], s52 offen sc0 sc1\n" \
  "buffer_load_dwordx4 v[188:191], %[ao], %[rs], s52 offen offset:1024 sc0 sc1\n" \
  "buffer_load_dwordx4 v[128:131], %[ao], %[rs], s53 offen sc0 sc1\n" \
  "buffer_load_dwordx4 v[132:135], %[ao], %[rs], s53 offen offset:1024 sc0 sc1\n" \
  "buffer_load_dwordx4 v[136:139], %[ao], %[rs], s54 offen sc0 sc1\n" \
  "buffer_load_dwordx4 v[140:143], %[ao], %[rs], s54 offen offset:1024 sc0 sc1\n" \
  "buffer_load_dwordx4 v[144:147], %[ao], %[rs], s55 offen sc0 sc1\n" \
  "buffer_load_dwordx4 v[148:151], %[ao], %[rs], s55 offen offset:1024 sc0 sc1\n" \
  "s_waitcnt vmcnt(6)\n" \
  "v_min3_u32 v126, v184, v185, v186\n" \
  "v_min3_u32 v125, v187, v188, v189\n" \
  "v_min3_u32 v126, v126, v190, v191\n" \
  "v_min_u32 v126, v126, v125\n" \
  "v_cmp_eq_u32 vcc, 0, v126\n" \
  "s_cbranch_vccz Lfr19_%=\n" \
  "s_sleep 2\n" \
  "s_branch Lfs19_%=\n" \
  "Lfs20_%=:\n" \
  "s_add_u32 s66, s66, 1\n" \
  "s_and_b32 s64, s66, 0x3f\n" \
  "s_cbranch_scc1 Lfk20_%=\n" \
  "s_swappc_b64 s[70:71], s[50:51]\n" \
  "Lfk20_%=:\n" \
  "v_readlane_b32 s52, %[tab], 20\n" \
  "v_readlane_b32 s53, %[tab], 21\n" \
  "v_readlane_b32 s54, %[tab], 22\n" \
  "v_readlane_b32 s55, %[tab], 23\n" \
  "s_nop 4\n" \
  "buffer_load_dwordx4 v[128:131], %[ao], %[rs], s52 offen sc0 sc1\n" \
  "buffer_load_dwordx4 v[132:135], %[ao], %[rs], s52 offen offset:1024 sc0 sc1\n" \
  "buffer_load_dwordx4 v[136:139], %[ao], %[rs], s53 offen sc0 sc1\n" \
  "buffer_load_dwordx4 v[140:143], %[ao], %[rs], s53 offen offset:1024 sc0 sc1\n" \
  "buffer_load_dwordx4 v[144:147], %[ao], %[rs], s54 offen sc0 sc1\n" \
  "buffer_load_dwordx4 v[148:151], %[ao], %[rs], s54 offen offset:1024 sc0 sc1\n" \
  "buffer_load_dwordx4 v[152:155], %[ao], %[rs], s55 offen sc0 sc1\n" \
  "buffer_load_dwordx4 v[156:159], %[ao], %[rs], s55 offen offset:1024 sc0 sc1\n" \
  "s_waitcnt vmcnt(6)\n" \
  "v_min3_u32 v126, v128, v129, v130\n" \
  "v_min3_u32 v125, v131, v132, v133\n" \
  "v_min3_u32 v126, v126, v134, v135\n" \
  "v_min_u32 v126, v126, v125\n" \
  "v_cmp_eq_u32 vcc, 0, v126\n" \
  "s_cbranch_vccz Lfr20_%=\n" \
  "s_sleep 2\n" \
  "s_branch Lfs20_%=\n" \
  "Lfs21_%=:\n" \
  "s_add_u32 s66, s66, 1\n" \
  "s_and_b32 s64, s66, 0x3f\n" \
  "s_cbranch_scc1 Lfk21_%=\n" \
  "s_swappc_b64 s[70:71], s[50:51]\n" \
  "Lfk21_%=:\n" \
  "v_readlane_b32 s52, %[tab], 21\n" \
  "v_readlane_b32 s53, %[tab], 22\n" \
  "v_readlane_b32 s54, %[tab], 23\n" \
  "v_readlane_b32 s55, %[tab], 24\n" \
  "s_nop 4\n" \
  "buffer_load_dwordx4 v[136:139], %[ao], %[rs], s52 offen sc0 sc1\n" \
  "buffer_load_dwordx4 v[140:143], %[ao], %[rs], s52 offen offset:1024 sc0 sc1\n" \
  "buffer_load_dwordx4 v[144:147], %[ao], %[rs], s53 offen sc0 sc1\n" \
  "buffer_load_dwordx4 v[148:151], %[ao], %[rs], s53 offen offset:1024 sc0 sc1\n" \
  "buffer_load_dwordx4 v[152:155], %[ao], %[rs], s54 offen sc0 sc1\n" \
  "buffer_load_dwordx4 v[156:159], %[ao], %[rs], s54 offen offset:1024 sc0 sc1\n" \
  "buffer_load_dwordx4 v[16:19], %[ao], %[rs], s55 offen sc0 sc1\n" \
  "buffer_load_dwordx4 v[20:23], %[ao], %[rs], s55 offen offset:1024 sc0 sc1\n" \
  "s_waitcnt vmcnt(6)\n" \
  "v_min3_u32 v126, v136, v137, v138\n" \
  "v_min3_u32 v125, v139, v140, v141\n" \
  "v_min3_u32 v126, v126, v142, v143\n" \
  "v_min_u32 v126, v126, v125\n" \
  "v_cmp_eq_u32 vcc, 0, v126\n" \
  "s_cbranch_vccz Lfr21_%=\n" \
  "s_sleep 2\n" \
  "s_branch Lfs21_%=\n" \
  "Lfs22_%=:\n" \
  "s_add_u32 s66, s66, 1\n" \
  "s_and_b32 s64, s66, 0x3f\n" \
  "s_cbranch_scc1 Lfk22_%=\n" \
  "s_swappc_b64 s[70:71], s[50:51]\n" \
  "Lfk22_%=:\n" \
  "v_readlane_b32 s52, %[tab], 22\n" \
  "v_readlane_b32 s53, %[tab], 23\n" \
  "v_readlane_b32 s54, %[tab], 24\n" \
  "v_readlane_b32 s55, %[tab], 25\n" \
  "s_nop 4\n" \
  "buffer_load_dwordx4 v[144:147], %[ao], %[rs], s52 offen sc0 sc1\n" \
  "buffer_load_dwordx4 v[148:151], %[ao], %[rs], s52 offen offset:1024 sc0 sc1\n" \
  "buffer_load_dwordx4 v[152:155], %[ao], %[rs], s53 offen sc0 sc1\n" \
  "buffer_load_dwordx4 v[156:159], %[ao], %[rs], s53 offen offset:1024 sc0 sc1\n" \
  "buffer_load_dwordx4 v[16:19], %[ao], %[rs], s54 offen sc0 sc1\n" \
  "buffer_load_dwordx4 v[20:23], %[ao], %[rs], s54 offen offset:1024 sc0 sc1\n" \
  "buffer_load_dwordx4 v[24:27], %[ao], %[rs], s55 offen sc0 sc1\n" \
  "buffer_load_dwordx4 v[28:31], %[ao], %[rs], s55 offen offset:1024 sc0 sc1\n" \
  "s_waitcnt vmcnt(6)\n" \
  "v_min3_u32 v126, v144, v145, v146\n" \
  "v_min3_u32 v125, v147, v148, v149\n" \
  "v_min3_u32 v126, v126, v150, v151\n" \
  "v_min_u32 v126, v126, v125\n" \
  "v_cmp_eq_u32 vcc, 0, v126\n" \
  "s_cbranch_vccz Lfr22_%=\n" \
  "s_sleep 2\n" \
  "s_branch Lfs22_%=\n" \
  "Lfs23_%=:\n" \
  "s_add_u32 s66, s66, 1\n" \
  "s_and_b32 s64, s66, 0x3f\n" \
  "s_cbranch_scc1 Lfk23_%=\n" \
  "s_swappc_b64 s[70:71], s[50:51]\n" \
  "Lfk23_%=:\n" \
  "v_readlane_b32 s52, %[tab], 23\n" \
  "v_readlane_b32 s53, %[tab], 24\n" \
  "v_readlane_b32 s54, %[tab], 25\n" \
  "v_readlane_b32 s55, %[tab], 26\n" \
  "s_nop 4\n" \
  "buffer_load_dwordx4 v[152:155], %[ao], %[rs], s52 offen sc0 sc1\n" \
  "buffer_load_dwordx4 v[156:159], %[ao], %[rs], s52 offen offset:1024 sc0 sc1\n" \
  "buffer_load_dwordx4 v[16:19], %[ao], %[rs], s53 offen sc0 sc1\n" \
  "buffer_load_dwordx4 v[20:23], %[ao], %[rs], s53 offen offset:1024 sc0 sc1\n" \
  "buffer_load_dwordx4 v[24:27], %[ao], %[rs], s54 offen sc0 sc1\n" \
  "buffer_load_dwordx4 v[28:31], %[ao], %[rs], s54 offen offset:1024 sc0 sc1\n" \
  "buffer_load_dwordx4 v[32:35], %[ao], %[rs], s55 offen sc0 sc1\n" \
  "buffer_load_dwordx4 v[36:39], %[ao], %[rs], s55 offen offset:1024 sc0 sc1\n" \
  "s_waitcnt vmcnt(6)\n" \
  "v_min3_u32 v126, v152, v153, v154\n" \
  "v_min3_u32 v125, v155, v156, v157\n" \
  "v_min3_u32 v126, v126, v158, v159\n" \
  "v_min_u32 v126, v126, v125\n" \
  "v_cmp_eq_u32 vcc, 0, v126\n" \
  "s_cbranch_vccz Lfr23_%=\n" \
  "s_sleep 2\n" \
  "s_branch Lfs23_%=\n" \
  "Lfs24_%=:\n" \
  "s_add_u32 s66, s66, 1\n" \
  "s_and_b32 s64, s66, 0x3f\n" \
  "s_cbranch_scc1 Lfk24_%=\n" \
  "s_swappc_b64 s[70:71], s[50:51]\n" \
  "Lfk24_%=:\n" \
  "v_readlane_b32 s52, %[tab], 24\n" \
  "v_readlane_b32 s53, %[tab], 25\n" \
  "v_readlane_b32 s54, %[tab], 26\n" \
  "v_readlane_b32 s55, %[tab], 27\n" \
  "s_nop 4\n" \
  "buffer_load_dwordx4 v[16:19], %[ao], %[rs], s52 offen sc0 sc1\n" \
  "buffer_load_dwordx4 v[20:23], %[ao], %[rs], s52 offen offset:1024 sc0 sc1\n" \
  "buffer_load_dwordx4 v[24:27], %[ao], %[rs], s53 offen sc0 sc1\n" \
  "buffer_load_dwordx4 v[28:31], %[ao], %[rs], s53 offen offset:1024 sc0 sc1\n" \
  "buffer_load_dwordx4 v[32:35], %[ao], %[rs], s54 offen sc0 sc1\n" \
  "buffer_load_dwordx4 v[36:39], %[ao], %[rs], s54 offen offset:1024 sc0 sc1\n" \
  "buffer_load_dwordx4 v[40:43], %[ao], %[rs], s55 offen sc0 sc1\n" \
  "buffer_load_dwordx4 v[44:47], %[ao], %[rs], s55 offen offset:1024 sc0 sc1\n" \
  "s_waitcnt vmcnt(6)\n" \
  "v_min3_u32 v126, v16, v17, v18\n" \
  "v_min3_u32 v125, v19, v20, v21\n" \
  "v_min3_u32 v126, v126, v22, v23\n" \
  "v_min_u32 v126, v126, v125\n" \
  "v_cmp_eq_u32 vcc, 0, v126\n" \
  "s_cbranch_vccz Lfr24_%=\n" \
  "s_sleep 2\n" \
  "s_branch Lfs24_%=\n" \
  "Lfs25_%=:\n" \
  "s_add_u32 s66, s66, 1\n" \
  "s_and_b32 s64, s66, 0x3f\n" \
  "s_cbranch_scc1 Lfk25_%=\n" \
  "s_swappc_b64 s[70:71], s[50:51]\n" \
  "Lfk25_%=:\n" \
  "v_readlane_b32 s52, %[tab], 25\n" \
  "v_readlane_b32 s53, %[tab], 26\n" \
  "v_readlane_b32 s54, %[tab], 27\n" \
  "v_readlane_b32 s55, %[tab], 30\n" \
  "s_nop 4\n" \
  "buffer_load_dwordx4 v[24:27], %[ao], %[rs], s52 offen sc0 sc1\n" \
  "buffer_load_dwordx4 v[28:31], %[ao], %[rs], s52 offen offset:1024 sc0 sc1\n" \
  "buffer_load_dwordx4 v[32:35], %[ao], %[rs], s53 offen sc0 sc1\n" \
  "buffer_load_dwordx4 v[36:39], %[ao], %[rs], s53 offen offset:1024 sc0 sc1\n" \
  "buffer_load_dwordx4 v[40:43], %[ao], %[rs], s54 offen sc0 sc1\n" \
  "buffer_load_dwordx4 v[44:47], %[ao], %[rs], s54 offen offset:1024 sc0 sc1\n" \
  "buffer_load_dwordx4 v[176:179], %[ao], %[rs], s55 offen sc0 sc1\n" \
  "buffer_load_dwordx4 v[180:183], %[ao], %[rs], s55 offen offset:1024 sc0 sc1\n" \
  "s_waitcnt vmcnt(6)\n" \
  "v_min3_u32 v126, v24, v25, v26\n" \
  "v_min3_u32 v125, v27, v28, v29\n" \
  "v_min3_u32 v126, v126, v30, v31\n" \
  "v_min_u32 v126, v126, v125\n" \
  "v_cmp_eq_u32 vcc, 0, v126\n" \
  "s_cbranch_vccz Lfr25_%=\n" \
  "s_sleep 2\n" \
  "s_branch Lfs25_%=\n" \
  "Lfs26_%=:\n" \
  "s_add_u32 s66, s66, 1\n" \
  "s_and_b32 s64, s66, 0x3f\n" \
  "s_cbranch_scc1 Lfk26_%=\n" \
  "s_swappc_b64 s[70:71], s[50:51]\n" \
  "Lfk26_%=:\n" \
  "v_readlane_b32 s52, %[tab], 26\n" \
  "v_readlane_b32 s53, %[tab], 27\n" \
  "v_readlane_b32 s54, %[tab], 30\n" \
  "v_readlane_b32 s55, %[tab], 31\n" \
  "s_nop 4\n" \
  "buffer_load_dwordx4 v[32:35], %[ao], %[rs], s52 offen sc0 sc1\n" \
  "buffer_load_dwordx4 v[36:39], %[ao], %[rs], s52 offen offset:1024 sc0 sc1\n" \
  "buffer_load_dwordx4 v[40:43], %[ao], %[rs], s53 offen sc0 sc1\n" \
  "buffer_load_dwordx4 v[44:47], %[ao], %[rs], s53 offen offset:1024 sc0 sc1\n" \
  "buffer_load_dwordx4 v[176:179], %[ao], %[rs], s54 offen sc0 sc1\n" \
  "buffer_load_dwordx4 v[180:183], %[ao], %[rs], s54 offen offset:1024 sc0 sc1\n" \
  "buffer_load_dwordx4 v[184:187], %[ao], %[rs], s55 offen sc0 sc1\n" \
  "buffer_load_dwordx4 v[188:191], %[ao], %[rs], s55 offen offset:1024 sc0 sc1\n" \
  "s_waitcnt vmcnt(6)\n" \
  "v_min3_u32 v126, v32, v33, v34\n" \
  "v_min3_u32 v125, v35, v36, v37\n" \
  "v_min3_u32 v126, v126, v38, v39\n" \
  "v_min_u32 v126, v126, v125\n" \
  "v_cmp_eq_u32 vcc, 0, v126\n" \
  "s_cbranch_vccz Lfr26_%=\n" \
  "s_sleep 2\n" \
  "s_branch Lfs26_%=\n" \
  "Lfs27_%=:\n" \
  "s_add_u32 s66, s66, 1\n" \
  "s_and_b32 s64, s66, 0x3f\n" \
  "s_cbranch_scc1 Lfk27_%=\n" \
  "s_swappc_b64 s[70:71], s[50:51]\n" \
  "Lfk27_%=:\n" \
  "v_readlane_b32 s52, %[tab], 27\n" \
  "v_readlane_b32 s53, %[tab], 30\n" \
  "v_readlane_b32 s54, %[tab], 31\n" \
  "v_readlane_b32 s55, %[tab], 28\n" \
  "s_nop 4\n" \
  "buffer_load_dwordx4 v[40:43], %[ao], %[rs], s52 offen sc0 sc1\n" \
  "buffer_load_dwordx4 v[44:47], %[ao], %[rs], s52 offen offset:1024 sc0 sc1\n" \
  "buffer_load_dwordx4 v[176:179], %[ao], %[rs], s53 offen sc0 sc1\n" \
  "buffer_load_dwordx4 v[180:183], %[ao], %[rs], s53 offen offset:1024 sc0 sc1\n" \
  "buffer_load_dwordx4 v[184:187], %[ao], %[rs], s54 offen sc0 sc1\n" \
  "buffer_load_dwordx4 v[188:191], %[ao], %[rs], s54 offen offset:1024 sc0 sc1\n" \
  "buffer_load_dwordx4 v[160:163], %[ao], %[rs], s55 offen sc0 sc1\n" \
  "buffer_load_dwordx4 v[164:167], %[ao], %[rs], s55 offen offset:1024 sc0 sc1\n" \
  "s_waitcnt vmcnt(6)\n" \
  "v_min3_u32 v126, v40, v41, v42\n" \
  "v_min3_u32 v125, v43, v44, v45\n" \
  "v_min3_u32 v126, v126, v46, v47\n" \
  "v_min_u32 v126, v126, v125\n" \
  "v_cmp_eq_u32 vcc, 0, v126\n" \
  "s_cbranch_vccz Lfr27_%=\n" \
  "s_sleep 2\n" \
  "s_branch Lfs27_%=\n" \
  "Lfs30_%=:\n" \
  "s_add_u32 s66, s66, 1\n" \
  "s_and_b32 s64, s66, 0x3f\n" \
  "s_cbranch_scc1 Lfk30_%=\n" \
  "s_swappc_b64 s[70:71], s[50:51]\n" \
  "Lfk30_%=:\n" \
  "v_readlane_b32 s52, %[tab], 30\n" \
  "v_readlane_b32 s53, %[tab], 31\n" \
  "v_readlane_b32 s54, %[tab], 28\n" \
  "v_readlane_b32 s55, %[tab], 29\n" \
  "s_nop 4\n" \
  "buffer_load_dwordx4 v[176:179], %[ao], %[rs], s52 offen sc0 sc1\n" \
  "buffer_load_dwordx4 v[180:183], %[ao], %[rs], s52 offen offset:1024 sc0 sc1\n" \
  "buffer_load_dwordx4 v[184:187], %[ao], %[rs], s53 offen sc0 sc1\n" \
  "buffer_load_dwordx4 v[188:191], %[ao], %[rs], s53 offen offset:1024 sc0 sc1\n" \
  "buffer_load_dwordx4 v[160:163], %[ao], %[rs], s54 offen sc0 sc1\n" \
  "buffer_load_dwordx4 v[164:167], %[ao], %[rs], s54 offen offset:1024 sc0 sc1\n" \
  "buffer_load_dwordx4 v[168:171], %[ao], %[rs], s55 offen sc0 sc1\n" \
  "buffer_load_dwordx4 v[172:175], %[ao], %[rs], s55 offen offset:1024 sc0 sc1\n" \
  "s_waitcnt vmcnt(6)\n" \
  "v_min3_u32 v126, v176, v177, v178\n" \
  "v_min3_u32 v125, v179, v180, v181\n" \
  "v_min3_u32 v126, v126, v182, v183\n" \
  "v_min_u32 v126, v126, v125\n" \
  "v_cmp_eq_u32 vcc, 0, v126\n" \
  "s_cbranch_vccz Lfr30_%=\n" \
  "s_sleep 2\n" \
  "s_branch Lfs30_%=\n" \
  "Lfs31_%=:\n" \
  "s_add_u32 s66, s66, 1\n" \
  "s_and_b32 s64, s66, 0x3f\n" \
  "s_cbranch_scc1 Lfk31_%=\n" \
  "s_swappc_b64 s[70:71], s[50:51]\n" \
  "Lfk31_%=:\n" \
  "v_readlane_b32 s52, %[tab], 31\n" \
  "v_readlane_b32 s53, %[tab], 28\n" \
  "v_readlane_b32 s54, %[tab], 29\n" \
  "s_nop 4\n" \
  "buffer_load_dwordx4 v[184:187], %[ao], %[rs], s52 offen sc0 sc1\n" \
  "buffer_load_dwordx4 v[188:191], %[ao], %[rs], s52 offen offset:1024 sc0 sc1\n" \
  "buffer_load_dwordx4 v[160:163], %[ao], %[rs], s53 offen sc0 sc1\n" \
  "buffer_load_dwordx4 v[164:167], %[ao], %[rs], s53 offen offset:1024 sc0 sc1\n" \
  "buffer_load_dwordx4 v[168:171], %[ao], %[rs], s54 offen sc0 sc1\n" \
  "buffer_load_dwordx4 v[172:175], %[ao], %[rs], s54 offen offset:1024 sc0 sc1\n" \
  "s_waitcnt vmcnt(4)\n" \
  "v_min3_u32 v126, v184, v185, v186\n" \
  "v_min3_u32 v125, v187, v188, v189\n" \
  "v_min3_u32 v126, v126, v190, v191\n" \
  "v_min_u32 v126, v126, v125\n" \
  "v_cmp_eq_u32 vcc, 0, v126\n" \
  "s_cbranch_vccz Lfr31_%=\n" \
  "s_sleep 2\n" \
  "s_branch Lfs31_%=\n" \
  "Lfs28_%=:\n" \
  "s_add_u32 s66, s66, 1\n" \
  "s_and_b32 s64, s66, 0x3f\n" \
  "s_cbranch_scc1 Lfk28_%=\n" \
  "s_swappc_b64 s[70:71], s[50:51]\n" \
  "Lfk28_%=:\n" \
  "v_readlane_b32 s52, %[tab], 28\n" \
  "v_readlane_b32 s53, %[tab], 29\n" \
  "s_nop 4\n" \
  "buffer_load_dwordx4 v[160:163], %[ao], %[rs], s52 offen sc0 sc1\n" \
  "buffer_load_dwordx4 v[164:167], %[ao], %[rs], s52 offen offset:1024 sc0 sc1\n" \
  "buffer_load_dwordx4 v[168:171], %[ao], %[rs], s53 offen sc0 sc1\n" \
  "buffer_load_dwordx4 v[172:175], %[ao], %[rs], s53 offen offset:1024 sc0 sc1\n" \
  "s_waitcnt vmcnt(2)\n" \
  "v_min3_u32 v126, v160, v161, v162\n" \
  "v_min3_u32 v125, v163, v164, v165\n" \
  "v_min3_u32 v126, v126, v166, v167\n" \
  "v_min_u32 v126, v126, v125\n" \
  "v_cmp_eq_u32 vcc, 0, v126\n" \
  "s_cbranch_vccz Lfr28_%=\n" \
  "s_sleep 2\n" \
  "s_branch Lfs28_%=\n" \
  "Lfs29_%=:\n" \
  "s_add_u32 s66, s66, 1\n" \
  "s_and_b32 s64, s66, 0x3f\n" \
  "s_cbranch_scc1 Lfk29_%=\n" \
  "s_swappc_b64 s[70:71], s[50:51]\n" \
  "Lfk29_%=:\n" \
  "v_readlane_b32 s52, %[tab], 29\n" \
  "s_nop 4\n" \
  "buffer_load_dwordx4 v[168:171], %[ao], %[rs], s52 offen sc0 sc1\n" \
  "buffer_load_dwordx4 v[172:175], %[ao], %[rs], s52 offen offset:1024 sc0 sc1\n" \
  "s_waitcnt vmcnt(0)\n" \
  "v_min3_u32 v126, v168, v169, v170\n" \
  "v_min3_u32 v125, v171, v172, v173\n" \
  "v_min3_u32 v126, v126, v174, v175\n" \
  "v_min_u32 v126, v126, v125\n" \
  "v_cmp_eq_u32 vcc, 0, v126\n" \
  "s_cbranch_vccz Lfr29_%=\n" \
  "s_sleep 2\n" \
  "s_branch Lfs29_%=\n" \
  "Ldone_%=:\n" \
  "v_mov_b32 v127, 99\n" \
  "ds_write_b32 %[dn], v127\n" \
  "s_waitcnt vmcnt(0) lgkmcnt(0)\n" \
  "s_nop 15\n" \
  "s_nop 7\n"
#define LOADER_ASM \
  "s_mov_b32 s60, 1\n" \
  "s_mov_b32 s64, 0\n" \
  "s_add_u32 s65, %[rb], 0x4000\n" \
  "s_barrier\n" \
  "s_cmp_lg_u32 %[hd], 0\n" \
  "s_cbranch_scc1 Lidle_%=\n" \
  "Ltop_%=:\n" \
  "s_sub_u32 s63, s60, 1\n" \
  "v_readlane_b32 s61, %[tab], s63\n" \
  "v_readlane_b32 s68, %[tab], s60\n" \
  "s_mov_b32 m0, s65\n" \
  "s_add_u32 s60, s60, 2\n" \
  "s_add_u32 s66, s65, 0x10000\n" \
  "s_add_u32 s65, s65, 0x2000\n" \
  "s_and_b32 s65, s65, 0xffff\n" \
  "buffer_load_dwordx4 %[o0], %[rs], s61 offen offset:0 lds\n" \
  "buffer_load_dwordx4 %[o1], %[rs], s61 offen offset:1024 lds\n" \
  "buffer_load_dwordx4 %[o2], %[rs], s61 offen offset:2048 lds\n" \
  "buffer_load_dwordx4 %[o3], %[rs], s61 offen offset:3072 lds\n" \
  "s_mov_b32 m0, s66\n" \
  "s_nop 0\n" \
  "buffer_load_dwordx4 %[o0], %[rs], s68 offen offset:0 lds\n" \
  "buffer_load_dwordx4 %[o1], %[rs], s68 offen offset:1024 lds\n" \
  "buffer_load_dwordx4 %[o2], %[rs], s68 offen offset:2048 lds\n" \
  "buffer_load_dwordx4 %[o3], %[rs], s68 offen offset:3072 lds\n" \
  "s_cmp_lg_u32 s60, 17\n" \
  "s_cbranch_scc1 Lidle_%=\n" \
  "v_readlane_b32 s61, %[tab], 30\n" \
  "v_readlane_b32 s68, %[tab], 31\n" \
  "s_add_u32 m0, %[rb], 140288\n" \
  "s_nop 3\n" \
  "buffer_load_dwordx4 %[o0], %[rs], s61 offen offset:0 lds\n" \
  "buffer_load_dwordx4 %[o1], %[rs], s61 offen offset:1024 lds\n" \
  "buffer_load_dwordx4 %[o2], %[rs], s61 offen offset:2048 lds\n" \
  "buffer_load_dwordx4 %[o3], %[rs], s61 offen offset:3072 lds\n" \
  "s_add_u32 m0, %[rb], 148480\n" \
  "s_nop 0\n" \
  "buffer_load_dwordx4 %[o0], %[rs], s68 offen offset:0 lds\n" \
  "buffer_load_dwordx4 %[o1], %[rs], s68 offen offset:1024 lds\n" \
  "buffer_load_dwordx4 %[o2], %[rs], s68 offen offset:2048 lds\n" \
  "buffer_load_dwordx4 %[o3], %[rs], s68 offen offset:3072 lds\n" \
  "Lidle_%=:\n" \
  "ds_read_b128 v[24:27], %[dn]\n" \
  "ds_read_b32 v20, %[dn] offset:16\n" \
  "s_getreg_b32 s67, hwreg(HW_REG_IB_STS)\n" \
  "s_bfe_u32 s63, s67, 0x20016\n" \
  "s_and_b32 s67, s67, 15\n" \
  "s_lshl4_add_u32 s67, s63, s67\n" \
  "s_add_u32 s67, s67, 7\n" \
  "s_lshr_b32 s67, s67, 3\n" \
  "s_lshl_b32 s63, s67, 1\n" \
  "s_sub_i32 s63, s60, s63\n" \
  "s_cmp_ge_i32 s63, 31\n" \
  "s_cselect_b32 s63, 99, s63\n" \
  "v_mov_b32 v21, s63\n" \
  "v_add_u32 v21, 2, v21\n" \
  "ds_write_b32 %[ld], v21\n" \
  "s_cbranch_scc1 Lwend_%=\n" \
  "s_add_u32 s64, s64, 1\n" \
  "s_and_b32 s63, s64, 0xff\n" \
  "s_cbranch_scc0 Lwtick_%=\n" \
  "Lidle2_%=:\n" \
  "s_cmp_lt_u32 s60, 16\n" \
  "s_cselect_b32 s63, 5, 3\n" \
  "s_waitcnt lgkmcnt(0)\n" \
  "v_min3_u32 v24, v24, v25, v20\n" \
  "v_min3_u32 v24, v24, v26, v27\n" \
  "s_cmp_gt_u32 s67, s63\n" \
  "v_readfirstlane_b32 s62, v24\n" \
  "s_cselect_b32 s62, 0, s62\n" \
  "s_cmp_lt_u32 s60, s62\n" \
  "s_cbranch_scc1 Ltop_%=\n" \
  "s_sleep 2\n" \
  "s_branch Lidle_%=\n" \
  "Lwtick_%=:\n" \
  "s_memrealtime s[74:75]\n" \
  "s_waitcnt lgkmcnt(0)\n" \
  "v_mov_b32 v22, s74\n" \
  "v_mov_b32 v23, s75\n" \
  "v_cmp_lt_u64 vcc, %[td], v[22:23]\n" \
  "s_cbranch_vccz Lidle2_%=\n" \
  "Lwend_%=:\n" \
  "s_waitcnt vmcnt(0) lgkmcnt(0)\n"
#define LOADER_CLOBBERS "v20", "v21", "v22", "v23", "v24", "v25", "v26", "v27", "s60", "s61", "s62", "s63", "s64", "s65", "s66", "s67", "s68", "s69", "s70", "s71", "s72", "s73", "s74", "s75"
#define MAIN_LOOP_CLOBBERS \
  "v12", "v13", "v14", "v15", "v16", "v17", "v18", "v19", "v20", "v21", "v22", "v23", "v24", "v25", "v26", \
  "v27", "v28", "v29", "v30", "v31", "v32", "v33", "v34", "v35", "v36", "v37", "v38", "v39", "v40", "v41", \
  "v42", "v43", "v44", "v45", "v46", "v47", "v112", "v113", "v114", "v115", "v116", "v117", "v118", "v119", \
  "v120", "v121", "v122", "v123", "v124", "v125", "v126", "v127", "v128", "v129", "v130", "v131", "v132", \
  "v133", "v134", "v135", "v136", "v137", "v138", "v139", "v140", "v141", "v142", "v143", "v144", "v145", \
  "v146", "v147", "v148", "v149", "v150", "v151", "v152", "v153", "v154", "v155", "v156", "v157", "v158", \
  "v159", "v160", "v161", "v162", "v163", "v164", "v165", "v166", "v167", "v168", "v169", "v170", "v171", \
  "v172", "v173", "v174", "v175", "v176", "v177", "v178", "v179", "v180", "v181", "v182", "v183", "v184", \
  "v185", "v186", "v187", "v188", "v189", "v190", "v191", "v192", "v193", "v194", "v195", "v196", "v197", \
  "v198", "v199", "v200", "v201", "v202", "v203", "v204", "v205", "v206", "v207", "v208", "v209", "v210", \
  "v211", "v212", "v213", "v214", "v215", "v216", "v217", "v218", "v219", "v220", "v221", "v222", "v223", \
  "v224", "v225", "v226", "v227", "v228", "v229", "v230", "v231", "v232", "v233", "v234", "v235", "v236", \
  "v237", "v238", "v239", "v240", "v241", "v242", "v243", "v244", "v245", "v246", "v247", "v248", "v249", \
  "v250", "v251", "v252", "v253", "v254", "v255", "s50", "s51", "s52", "s53", "s54", "s55", "s56", "s57", "s58", \
  "s59", "s60", "s61", "s62", "s63", "s64", "s65", "s66", "s67", "s68", "s69", "s70", "s71", "s72", "s73", \
  "s74", "s75", "s76", "s77"

__global__ void __launch_bounds__(512, 2) fused_a2a_gemm(Params p) {
  extern __shared__ __attribute__((aligned(16))) unsigned char lds_raw[];
  LAS unsigned char* L8 = (LAS unsigned char*)lds_raw;
  volatile LAS unsigned* SY = (volatile LAS unsigned*)(L8 + SYNC_OFF);
  const unsigned long long t0 = rt();
  asm volatile("" :: "s"(t0), "s"(p.x), "s"(p.w), "s"(p.myws), "s"(p.rank), "s"(p.sx), "s"(p.sw), "s"(p.out));
  const int b = blockIdx.x;
  const int wid = __builtin_amdgcn_readfirstlane(threadIdx.x >> 6), lane = threadIdx.x & 63;
  const int rank = p.rank & 7;
  unsigned char* myws = p.myws;
  const int xcd = b & 7, idx = b >> 3;
  const int kp = idx & 1, nt = xcd * 4 + ((idx >> 1) & 3), mt = idx >> 3;
  const int cw = wid & 3;
  const int di = idx & 7, jp = idx >> 3, pm = xcd;
  const int ms = jp * 8 + 2 * (((pm >> 1) + di) & 3) + (pm & 1);
  const int hold = ms < T_HOLD ? di : 0;
  const unsigned long long tdead = t0 + DEADLINE;
  const unsigned dummy = SYNC_OFF + 256 + lane * 4;

  float sxv, swv;
  auto load_scales = [&]() {
    asm volatile("s_load_dword %0, %2, 0x0\n\ts_load_dword %1, %3, 0x0\n\ts_waitcnt lgkmcnt(0)" : "=&s"(sxv), "=&s"(swv) : "s"(p.sx), "s"(p.sw) : "memory");
  };
  int badi = 0;
  v16f own, oth;
  const unsigned park = (cw >> 1) * RING_B + STAGE_B + (cw & 1) * 4096;
  const auto rsQ = __builtin_amdgcn_make_buffer_rsrc((void*)(myws + PRT_OFF + (b * 4 + cw) * 8192), 0, 8192, 0x00020000);

  if (wid == 2 || wid == 3) {
    const int par = wid - 2;
    unsigned vo[4];
#pragma unroll
    for (int i = 0; i < 4; i++)
      vo[i] = (lane >> 3) * 4096 + (((lane & 7) ^ ((lane >> 3) & 6)) * 16) + (par * 4 + i) * 32768 - i * 1024;
    const int tab = kidx_of(lane & 31, kp, rank, mt) * (64 * 4096);
    const auto rsW = __builtin_amdgcn_make_buffer_rsrc((void*)(p.w + nt * 128), 0, 4096 * 4096 - nt * 128, 0x00020000);
    const unsigned ld = lane == 0 ? SYNC_OFF + 4 * par : dummy;
    const unsigned dn = SYNC_OFF + 32;
    asm volatile(LOADER_ASM
                 :
                 : [o0] "v"(vo[0]), [o1] "v"(vo[1]), [o2] "v"(vo[2]), [o3] "v"(vo[3]), [tab] "v"(tab),
                   [ld] "v"(ld), [dn] "v"(dn), [rs] "s"(rsW), [td] "s"(tdead), [rb] "s"(par * 4096), [hd] "s"(hold)
                 : "memory", "vcc", "scc", LOADER_CLOBBERS);
  } else {

  const unsigned char* ctr = myws + CTR_OFF + b * 64;
  const unsigned zero_v = 0;
  unsigned callv;
  asm volatile("global_load_dword %0, %1, %2 sc1" : "=v"(callv) : "v"(zero_v), "s"(ctr) : "memory");
  if (wid != 1) {
    const unsigned char* src = p.x + (wid ? (rank * 512 + mt * 128 + cw * 32) * 512 + kp * 256 : (((rank + di) & 7) * 512 + pm * 64) * 512 + jp * 128);
    const int half = wid ? 128 : 4 * 4096;
    const unsigned dst = wid ? (cw >> 1) * RING_B + (cw & 1) * STAGE_B : SCR_OFF;
    const unsigned v0 = (lane >> 3) * 512 + (((lane & 7) ^ (lane >> 4)) * 16);
    const unsigned v1 = (v0 ^ 64) + 4096 - 1024, v2 = v0 + 8192 - 2048, v3 = (v0 ^ 64) + 12288 - 3072;
    asm volatile("s_mov_b32 m0, %6\n\ts_nop 0\n\t"
                 "global_load_lds_dwordx4 %0, %4\n\tglobal_load_lds_dwordx4 %1, %4 offset:1024\n\t"
                 "global_load_lds_dwordx4 %2, %4 offset:2048\n\tglobal_load_lds_dwordx4 %3, %4 offset:3072\n\t"
                 "s_add_u32 m0, %6, 0x1000\n\ts_nop 0\n\t"
                 "global_load_lds_dwordx4 %0, %5\n\tglobal_load_lds_dwordx4 %1, %5 offset:1024\n\t"
                 "global_load_lds_dwordx4 %2, %5 offset:2048\n\tglobal_load_lds_dwordx4 %3, %5 offset:3072"
                 :: "v"(v0), "v"(v1), "v"(v2), "v"(v3), "s"(src), "s"(src + half), "s"(dst) : "memory", "scc");
  } else {
    SY[lane] = (lane >= 8 && lane < 12) ? 12u : (lane == 12 && !hold) ? 30u : 0u;
    asm volatile("s_waitcnt lgkmcnt(0)" ::: "memory");
  }
  asm volatile("s_barrier" ::: "memory");

  if (wid == 1) {
    __builtin_amdgcn_s_setprio(3);
    {
      const unsigned c0 = (unsigned)((size_t)p.w >> 7);
      const unsigned v0 = (b * 16 + (lane >> 2)) * 4096 + (((3u - c0) & 7u) + 8 * (lane & 3)) * 128;
      const unsigned v1 = (b * 16 + (lane & 15)) * 4096 + ((31u - c0) & 31u) * 128;
      asm volatile("global_load_dword v254, %0, %2\n\tglobal_load_dword v255, %1, %2" :: "v"(v0), "v"(v1), "s"(p.w) : "memory", "v254", "v255");
    }
    const int kx = kidx_of(lane & 31, kp, rank, mt);
    const int tab = kx * 32768 + (2 * mt + 1) * 4096 + 3072 + ((kx >> 1) & 7) * 128;
    const unsigned call = call_number(callv);
    const unsigned char* pb = myws + STG_OFF + (call % 3u) * STG_B;
    const unsigned pa = lane == 0 ? SYNC_OFF + 8 : dummy;
    for (int avail = 4; avail < 32;) {
      unsigned o[7], v[7];
#pragma unroll
      for (int i = 0; i < 7; i++) o[i] = __builtin_amdgcn_readlane(tab, avail + i);
      asm volatile("s_nop 4\n\ts_load_dword %0, %7, %8 glc\n\ts_load_dword %1, %7, %9 glc\n\ts_load_dword %2, %7, %10 glc\n\t"
                   "s_load_dword %3, %7, %11 glc\n\ts_load_dword %4, %7, %12 glc\n\ts_load_dword %5, %7, %13 glc\n\t"
                   "s_load_dword %6, %7, %14 glc\n\ts_waitcnt lgkmcnt(0)"
                   : "=&s"(v[0]), "=&s"(v[1]), "=&s"(v[2]), "=&s"(v[3]), "=&s"(v[4]), "=&s"(v[5]), "=&s"(v[6])
                   : "s"(pb), "s"(o[0]), "s"(o[1]), "s"(o[2]), "s"(o[3]), "s"(o[4]), "s"(o[5]), "s"(o[6]) : "memory");
      int n = 7;
#pragma unroll
      for (int i = 6; i >= 0; i--) n = v[i] == 0u ? i : n;
      avail += n;
      asm volatile("ds_max_u32 %0, %1" :: "v"(pa), "v"(avail >= 32 ? 99 : avail) : "memory");
      if (n < 7) break;
    }
    const unsigned char* fb = myws + FLG_OFF + b * 128;
    for (int spins = 1;; spins++) {
      const v4u dn = *(volatile LAS v4u*)(SY + 8);
      const int far = __builtin_amdgcn_readfirstlane(max(max(dn.x, dn.y), max(dn.z, dn.w)));
      if (far >= 33) break;
      __builtin_amdgcn_s_sleep(8);
      if ((spins & 63) == 0 && rt() > tdead) break;
    }
    for (int spins = 1;; spins++) {
      v4u f4;
      asm volatile("s_load_dwordx4 %0, %1, 0x0 glc\n\ts_waitcnt lgkmcnt(0)" : "=s"(f4) : "s"(fb) : "memory");
      const unsigned f = lane == 0 ? f4.x : lane == 1 ? f4.y : lane == 2 ? f4.z : f4.w;
      const bool ok = (int)(f - call) >= 0;
      SY[lane < 4 ? 16 + lane : 64 + lane] = ok ? 1u : 0u;
      if (__builtin_amdgcn_ballot_w64(ok) == ~0ull) break;
      if ((spins & 63) == 0 && rt() > tdead) break;
    }
  } else
  if (wid == 0) {
    __builtin_amdgcn_s_setprio(3);
    const unsigned char* dst0 = p.wsr[di] + STG_OFF + (size_t)(rank * 8 + 2 * jp) * 32768 + pm * 4096;
    const unsigned dvo = lane * 16;
    const unsigned ra = SCR_OFF + (lane & 31) * 128 + ((((lane >> 5) * 2) ^ ((lane >> 1) & 7)) * 16);
    const int tb = T_TSTART + T_SLOT * ms;
    const int tn = lane < 4 ? 0 : lane >= 32 ? 0x7fffffff : max(T_TSTART + T_LAT + (slot_of(lane, mt) | 1) * T_SLOT, T_TMIN + 8 * lane);
    const unsigned pa = lane == 0 ? SYNC_OFF + 8 : dummy;
    auto tick_until = [&](int t) {
      for (;;) {
        const int now = (int)(rt() - t0);
        const int nom = __builtin_popcountll(__builtin_amdgcn_ballot_w64(tn <= now));
        asm volatile("ds_max_u32 %0, %1" :: "v"(pa), "v"(nom >= 32 ? 99 : nom) : "memory");
        if (now >= t) break;
        __builtin_amdgcn_s_sleep(1);
      }
    };
    asm volatile("" :: "s"(dst0));
    const unsigned call = call_number(callv);
    if (di != 0) {
      const unsigned char* dstb = dst0 + (call % 3u) * STG_B;
      v4u y[8];
#pragma unroll
      for (int i = 0; i < 8; i++) {
        v4u v = *(const LAS v4u*)(L8 + ((ra ^ ((i >> 2) * 64 + (i & 1) * 16)) + ((i >> 1) & 1) * 4096));
        y[i] = (v4u){nzfix(v.x), nzfix(v.y), nzfix(v.z), nzfix(v.w)};
      }
      tick_until(tb);
      asm volatile("global_store_dwordx4 %0, %1, %9 sc0 sc1\n\tglobal_store_dwordx4 %0, %2, %9 offset:1024 sc0 sc1\n\t"
                   "global_store_dwordx4 %0, %3, %9 offset:2048 sc0 sc1\n\tglobal_store_dwordx4 %0, %4, %9 offset:3072 sc0 sc1\n\t"
                   "global_store_dwordx4 %0, %5, %10 sc0 sc1\n\tglobal_store_dwordx4 %0, %6, %10 offset:1024 sc0 sc1\n\t"
                   "global_store_dwordx4 %0, %7, %10 offset:2048 sc0 sc1\n\tglobal_store_dwordx4 %0, %8, %10 offset:3072 sc0 sc1\n\ts_nop 1"
                   :: "v"(dvo), "v"(y[0]), "v"(y[1]), "v"(y[2]), "v"(y[3]), "v"(y[4]), "v"(y[5]), "v"(y[6]), "v"(y[7]),
                      "s"(dstb), "s"(dstb + 32768) : "memory");
      SY[lane == 0 ? 12 : 64 + lane] = 30u;
    }
    {
      const unsigned char* z = myws + STG_OFF + ((call + 2u) % 3u) * STG_B + b * 8192;
      const v4u zero = {0, 0, 0, 0};
      asm volatile("global_store_dwordx4 %0, %1, %2 sc0 sc1\n\tglobal_store_dwordx4 %0, %1, %2 offset:1024 sc0 sc1\n\t"
                   "global_store_dwordx4 %0, %1, %2 offset:2048 sc0 sc1\n\tglobal_store_dwordx4 %0, %1, %2 offset:3072 sc0 sc1\n\t"
                   "global_store_dwordx4 %0, %1, %3 sc0 sc1\n\tglobal_store_dwordx4 %0, %1, %3 offset:1024 sc0 sc1\n\t"
                   "global_store_dwordx4 %0, %1, %3 offset:2048 sc0 sc1\n\tglobal_store_dwordx4 %0, %1, %3 offset:3072 sc0 sc1\n\ts_nop 1"
                   :: "v"(dvo), "v"(zero), "s"(z), "s"(z + 4096) : "memory");
    }
    if (lane == 0) __hip_atomic_store((GAS unsigned*)ctr, call, __ATOMIC_RELAXED, __HIP_MEMORY_SCOPE_AGENT);
    tick_until(max(T_TSTART + T_LAT + 32 * T_SLOT, T_TMIN + 256));
  } else {
    __builtin_amdgcn_s_setprio(2);
    v16f acc[4];
    const int tab = kidx_of(lane & 31, kp, rank, mt) * 32768;
    const int avoff = lane * 16;
    const int bq = (lane & 15) >> 1;
    const unsigned baddr0 = (((lane >> 5) * 32 + bq) * 128 + (((bq & 6) + ((lane >> 4) & 1)) * 16) + (lane & 1) * 8) ^ (kp << 6);
    const unsigned lcl = (cw >> 1) * RING_B + (cw & 1) * STAGE_B + (lane & 31) * 128 + ((((lane >> 5) * 2) ^ ((lane >> 1) & 7)) * 16);
    const unsigned vsync = SYNC_OFF;
    const unsigned vdone = lane == 0 ? SYNC_OFF + 4 * (8 + cw) : dummy;
#pragma unroll
    for (int i = 0; i < 4; i++)
#pragma unroll
      for (int j = 0; j < 16; j++) acc[i][j] = 0.f;
    const auto rsP = __builtin_amdgcn_make_buffer_rsrc((void*)(myws + PRT_OFF + ((b ^ 8) * 4 + cw) * 8192), 0, 8192, 0x00020000);
    const auto rsG = __builtin_amdgcn_make_buffer_rsrc((void*)(myws + FLG_OFF + (b ^ 8) * 128 + cw * 4), 0, 4, 0x00020000);
    load_scales();
    const unsigned call = call_number(callv);
    const auto rsA = __builtin_amdgcn_make_buffer_rsrc((void*)(myws + STG_OFF + (call % 3u) * STG_B + (4 * mt + cw) * 2048), 0,
                                                       (int)STG_B - (4 * mt + cw) * 2048, 0x00020000);
    asm volatile(MAIN_LOOP_ASM
                 : "+{v[48:63]}"(acc[0]), "+{v[64:79]}"(acc[1]), "+{v[80:95]}"(acc[2]), "+{v[96:111]}"(acc[3]), "+s"(badi)
                 : [tab] "v"(tab), [ao] "v"(avoff), [b0] "v"(baddr0), [sy] "v"(vsync), [dn] "v"(vdone), [la] "v"(lcl),
                   [rs] "s"(rsA), [td] "s"(tdead), [rp] "s"(rsP), [rq] "s"(rsQ), [rf] "s"(rsG), [cl] "s"(call), [pk] "s"(park)
                 : "memory", "vcc", "scc", MAIN_LOOP_CLOBBERS);
    own = acc[0];
    oth = acc[2];
  }
  }
  if (wid < 4) {
    __builtin_amdgcn_s_setprio(1);
    load_scales();
    for (int i = 1; __builtin_amdgcn_readfirstlane(SY[16 + cw]) == 0; i++) {
      __builtin_amdgcn_s_sleep(1);
      if ((i & 255) == 0 && rt() > tdead) { badi = 1; break; }
    }
    asm volatile("" ::: "memory");
#pragma unroll
    for (int i = 0; i < 4; i++) {
      const v4u t = __builtin_amdgcn_raw_buffer_load_b128(rsQ, lane * 16 + i * 1024, 4096, 16);
      oth[4 * i] = __uint_as_float(t.x); oth[4 * i + 1] = __uint_as_float(t.y); oth[4 * i + 2] = __uint_as_float(t.z); oth[4 * i + 3] = __uint_as_float(t.w);
    }
    for (int i = 1; __builtin_amdgcn_readfirstlane(SY[20 + cw]) == 0; i++)
      if ((i & 255) == 0 && rt() > tdead) { badi = 1; break; }
    asm volatile("" ::: "memory");
#pragma unroll
    for (int i = 0; i < 4; i++) {
      const v4u t = *(const LAS v4u*)(L8 + park + i * 1024 + lane * 16);
      own[4 * i] = __uint_as_float(t.x); own[4 * i + 1] = __uint_as_float(t.y); own[4 * i + 2] = __uint_as_float(t.z); own[4 * i + 3] = __uint_as_float(t.w);
    }
  }
  const auto rsO = __builtin_amdgcn_make_buffer_rsrc((void*)(p.out + (size_t)(mt * 128 + cw * 32) * 4096 + nt * 128 + kp * 64 + (wid < 4 ? 32 : 0)),
                                                     0, 31 * 16384 + 128, 0x00020000);
  const int ovoff = (lane >> 5) * (4 * 16384) + (lane & 31) * 4;
  const float sc1 = badi ? __builtin_nanf("") : sxv * swv;
  const float sc2 = sc1 * -1.4426950408889634f;
  const float lim = 60.f * 1.4426950408889634f;
#pragma unroll
  for (int j = 0; j < 4; j++) {
#pragma unroll
    for (int e = 0; e < 4; e += 2) {
      v2f s = (v2f){own[4 * j + e], own[4 * j + e + 1]} + (v2f){oth[4 * j + e], oth[4 * j + e + 1]};
      v2f y = s * sc1;
      v2f z = s * sc2;
      v2f ex = {__builtin_amdgcn_exp2f(__builtin_amdgcn_fmed3f(z.x, -lim, lim)),
                __builtin_amdgcn_exp2f(__builtin_amdgcn_fmed3f(z.y, -lim, lim))};
      v2f dn = ex + 1.0f;
      v2f r = y * (v2f){__builtin_amdgcn_rcpf(dn.x), __builtin_amdgcn_rcpf(dn.y)};
      const float rx = r.x, ry = r.y;
      __builtin_amdgcn_raw_buffer_store_b32(__float_as_uint(rx), rsO, ovoff, (8 * j + e) * 16384, 16);
      __builtin_amdgcn_raw_buffer_store_b32(__float_as_uint(ry), rsO, ovoff, (8 * j + e + 1) * 16384, 16);
    }
  }
}


extern "C" void kernel_launch(int rank, int nranks, ncclComm_t comm,
                              hipStream_t stream,
                              void* const* d_in, const int* in_sizes, int n_in,
                              void* d_out, int out_size,
                              const int* in_dtypes, int out_dtype,
                              void* const* d_out_all,
                              void* const* d_ws_all, size_t ws_bytes) {
  if (nranks != NRANK || (unsigned)rank >= (unsigned)NRANK || ws_bytes < WS_NEED || n_in < 4 || in_sizes[0] != 4096 * 512 ||
      in_sizes[1] != 4096 * 4096 || out_size != 512 * 4096) {
    fprintf(stderr, "unsupported configuration\n");
    return;
  }
  Params p;
  p.x = (const unsigned char*)d_in[0];
  p.w = (const unsigned char*)d_in[1];
  p.sx = (const float*)d_in[2];
  p.sw = (const float*)d_in[3];
  p.out = (float*)d_out;
  for (int i = 0; i < NRANK; i++) p.wsr[i] = (unsigned char*)d_ws_all[(rank + i) & 7];
  p.myws = p.wsr[0];
  p.rank = rank;
  hipError_t e = hipFuncSetAttribute((const void*)fused_a2a_gemm, hipFuncAttributeMaxDynamicSharedMemorySize, LDS_B);
  if (e != hipSuccess) fprintf(stderr, "attr: %s\n", hipGetErrorString(e));
  hipLaunchKernelGGL(fused_a2a_gemm, dim3(256), dim3(512), LDS_B, stream, p);
}
```
